```python
import numpy as np

kernel_name = "gpu_ag_gemm_m4096_k4096_n4096_w8a8_dqsilu"


ABSMAX_RTOL = 0.001

COMPUTE_DTYPE = "int8"
EPILOGUE = "dequant_silu"
IN_QUANT = "int8"
OUT_QUANT = None
IN_DTYPES = {"x": "int8", "w_mat": "int8",
             "scale_x": "float32", "scale_w": "float32"}
WORLD_SIZE = 8

def setup_inputs():

    x = np.random.randint(-127, 128, size=(4096, 4096)).astype(np.int8)
    w_mat = np.random.randint(-127, 128, size=(4096, 4096)).astype(np.int8)
    scale_x = (0.02 * (1.0 + np.random.rand(1))).astype(np.float32)
    scale_w = (0.02 * (1.0 + np.random.rand(1))).astype(np.float32)
    return {"x": x, "w_mat": w_mat, "scale_x": scale_x, "scale_w": scale_w}

def reference(x, w_mat, scale_x, scale_w):
    acc = x.astype(np.float64) @ w_mat.astype(np.float64)
    y = (acc.astype(np.float32) * (scale_x * scale_w)).astype(np.float32)
    return (y / (1.0 + np.exp(-np.clip(y, -60.0, 60.0)))).astype(np.float32)
```

```cpp
#include <hip/hip_runtime.h>
#include <rccl/rccl.h>
#include <cstdio>

#define HIP_CHECK(expr) do { hipError_t _e = (expr); if (_e != hipSuccess) { \
    fprintf(stderr, "HIP error %s at %s:%d\n", hipGetErrorString(_e), __FILE__, __LINE__); \
    return; } } while (0)

#define SEND_W 16
#define SEND_P 8
#define A_AUX 17
#define O_AUX 17
#define Z_AUX 17
#define AGGR_RB 31
#define Z_DELAY 300
#define Z_PACE 8

namespace {

typedef int v4i __attribute__((ext_vector_type(4)));
typedef unsigned v4u __attribute__((ext_vector_type(4)));
typedef float v4f __attribute__((ext_vector_type(4)));

constexpr int NR = 8;
constexpr int M_SH = 512, K_DIM = 4096, N_SH = 512;
constexpr size_t SHARD = (size_t)M_SH * K_DIM;
constexpr int UNITS = 512;
constexpr size_t BUF_BYTES = (size_t)NR * SHARD;
constexpr size_t STAGE_OFF = 0;
constexpr size_t FLAG_OFF = 3 * BUF_BYTES;
constexpr size_t CTR_OFF = FLAG_OFF + (size_t)NR * 64 * 128;
constexpr int NWG = 256, NWAVE = 8;
constexpr size_t TMO_OFF = CTR_OFF + (size_t)NWG * NWAVE * 4;
constexpr size_t DBG_OFF = TMO_OFF + 128;
constexpr size_t DBG_WORDS = (size_t)NWG * NWAVE * 16;
constexpr size_t WS_NEED = DBG_OFF + DBG_WORDS * 8;
constexpr unsigned long long DEADLINE_TICKS = 100000000ull;
constexpr int NGROUP = 24;
constexpr int XORK = (int)0x80808080u;

struct Params {
  char* peer[2 * (NR - 1)];
  char* ws;
  const signed char* x;
  const signed char* w;
  const float* sx;
  const float* sw;
  float* out;
  int rank;
};

#define WAITVM(n) asm volatile("s_waitcnt vmcnt(%0)" ::"n"(n) : "memory")

template <int IMM>
__device__ __forceinline__ void st_stage(unsigned voff, v4i d, char* sbase) {
  asm volatile("global_store_dwordx4 %0, %1, %2 offset:%3 sc0 sc1" ::"v"(voff), "v"(d), "s"(sbase), "n"(IMM) : "memory");
}
template <int IMM>
__device__ __forceinline__ void ld_x(v4i& d, unsigned voff, const signed char* sbase) {
  asm volatile("global_load_dwordx4 %0, %1, %2 offset:%3" : "+v"(d) : "v"(voff), "s"(sbase), "n"(IMM) : "memory");
}

template <int F, int WCNT>
__device__ __forceinline__ void send_frag(char* const (&dst)[NR - 1], unsigned vdst, v4i& d, unsigned vs, const signed char* x) {
  asm volatile("s_waitcnt vmcnt(%1)" : "+v"(d) : "n"(WCNT) : "memory");
  d ^= XORK;
#pragma unroll
  for (int j = 0; j < NR - 1; ++j) st_stage<F * 1024>(vdst, d, dst[j]);
  ld_x<F * 64>(d, vs, x);
}

template <int W, int P>
__device__ __forceinline__ void sender(const Params& p, int v, int lane, unsigned* myctr) {
  constexpr int UPI = P / 4;
  constexpr int NU = UNITS / W;
  constexpr int NIT = NU / UPI;
  constexpr unsigned USTEP = (unsigned)W * 4096u;
  constexpr int WCNT = 8 * (P - 1) < 54 ? 8 * (P - 1) : 54;

  unsigned vsrc = (unsigned)((v >> 4) * 65536 + (v & 15) * 256 + (lane & 15) * 4096 + (lane >> 4) * 16);
  v4i buf[P];
#pragma unroll
  for (int s = 0; s < P; ++s) buf[s] = (v4i){0, 0, 0, 0};
  const unsigned cprev = __hip_atomic_load(myctr, __ATOMIC_RELAXED, __HIP_MEMORY_SCOPE_AGENT);
  {
    unsigned vs = vsrc;
#pragma unroll
    for (int uu = 0; uu < UPI; ++uu) {
      ld_x<0>(buf[uu * 4 + 0], vs, p.x);
      ld_x<64>(buf[uu * 4 + 1], vs, p.x);
      ld_x<128>(buf[uu * 4 + 2], vs, p.x);
      ld_x<192>(buf[uu * 4 + 3], vs, p.x);
      vs += USTEP;
    }
  }
  char* dst[NR - 1];
  {
    char* const* pp = p.peer + v % (NR - 1);
#pragma unroll
    for (int j = 0; j < NR - 1; ++j) dst[j] = pp[j];
    asm volatile("" : "+s"(dst[0]), "+s"(dst[1]), "+s"(dst[2]), "+s"(dst[3]), "+s"(dst[4]), "+s"(dst[5]), "+s"(dst[6]));
  }
  const unsigned call = __builtin_amdgcn_readfirstlane(cprev) + 1u;
  unsigned vdst = (unsigned)STAGE_OFF + (call % 3u) * (unsigned)BUF_BYTES + (unsigned)p.rank * (unsigned)SHARD +
                  (unsigned)(v * 4096 + lane * 16);
  WAITVM(0);

  for (int it = 0; it < NIT; ++it) {
    const bool more = (it + 1 < NIT);
#pragma unroll
    for (int uu = 0; uu < UPI; ++uu) {
      const unsigned vs = vsrc + (more ? (unsigned)(UPI + uu) * USTEP : 0u);
      const unsigned vd = vdst + (unsigned)uu * USTEP;
      send_frag<0, WCNT>(dst, vd, buf[uu * 4 + 0], vs, p.x);
      send_frag<1, WCNT>(dst, vd, buf[uu * 4 + 1], vs, p.x);
      send_frag<2, WCNT>(dst, vd, buf[uu * 4 + 2], vs, p.x);
      send_frag<3, WCNT>(dst, vd, buf[uu * 4 + 3], vs, p.x);
    }
    vsrc += (unsigned)UPI * USTEP;
    vdst += (unsigned)UPI * USTEP;
  }
  WAITVM(0);
#pragma unroll
  for (int s = 0; s < P; ++s) asm volatile("" : "+v"(buf[s])::"memory");
  if (lane == 0) {
    const size_t foff = FLAG_OFF + ((size_t)p.rank * 64 + v) * 128;
#pragma unroll
    for (int j = 0; j < NR - 1; ++j)
      __hip_atomic_store((unsigned*)(dst[j] + foff), call, __ATOMIC_RELAXED, __HIP_MEMORY_SCOPE_SYSTEM);
    __hip_atomic_store(myctr, call, __ATOMIC_RELAXED, __HIP_MEMORY_SCOPE_AGENT);
  }
}

__device__ __forceinline__ unsigned perm(unsigned hi, unsigned lo, unsigned sel) {
  return __builtin_amdgcn_perm(hi, lo, sel);
}

__device__ __forceinline__ float silu_dq(int acc, float scale) {
  const float y = (float)acc * scale;
  const float yc = fminf(fmaxf(y, -60.0f), 60.0f);
  return y * __builtin_amdgcn_rcpf(1.0f + __expf(-yc));
}

__device__ __forceinline__ unsigned umin3(unsigned a, unsigned b, unsigned c) { return min(min(a, b), c); }

__global__ void __launch_bounds__(512, 2) ag_gemm_kernel(Params p) {
  __shared__ v4i red[2][NWAVE][256];
  __shared__ int bad_flag;
  const int b = blockIdx.x, tid = threadIdx.x, lane = tid & 63;
  const int wave = __builtin_amdgcn_readfirstlane(tid >> 6);
  const int xcd = b & 7, idx = b >> 3;
  unsigned* myctr = (unsigned*)(p.ws + CTR_OFF) + b * NWAVE + wave;

  if (idx < 8) {
    if (wave == 0) {
      if (b < SEND_W) {
        sender<SEND_W, SEND_P>(p, b, lane, myctr);
      }
    } else {
      const unsigned long long tz = __builtin_amdgcn_s_memrealtime();
      const unsigned cprev = __hip_atomic_load(myctr, __ATOMIC_RELAXED, __HIP_MEMORY_SCOPE_AGENT);
      const unsigned call = __builtin_amdgcn_readfirstlane(cprev) + 1u;
      const __amdgpu_buffer_rsrc_t rsZ =
          __builtin_amdgcn_make_buffer_rsrc(p.ws + STAGE_OFF + (size_t)((call + 2u) % 3u) * BUF_BYTES, 0, (int)BUF_BYTES, 0x00020000);
      if (lane == 0) __hip_atomic_store(myctr, call, __ATOMIC_RELAXED, __HIP_MEMORY_SCOPE_AGENT);
      const int off = ((p.rank + wave) & 7) * (int)SHARD + b * 32768 + lane * 16;
      while (__builtin_amdgcn_s_memrealtime() - tz < Z_DELAY) __builtin_amdgcn_s_sleep(8);
      for (int i = 0; i < 32; ++i) {
        __builtin_amdgcn_raw_buffer_store_b128((v4u){0u, 0u, 0u, 0u}, rsZ, off + i * 1024, 0, Z_AUX);
        __builtin_amdgcn_s_sleep(Z_PACE);
      }
    }
    return;
  }

  const int cidx = idx - 8;
  const int grp = (cidx >> 3) * 8 + xcd;
  const int panel = cidx & 7;
  const int c = lane & 15, g = lane >> 4;
  const unsigned cprev = __hip_atomic_load(myctr, __ATOMIC_RELAXED, __HIP_MEMORY_SCOPE_AGENT);
  if (tid == 0) bad_flag = 0;

  v4i B[8][4];
  {
    const signed char* wp = p.w + (size_t)(wave * 512 + g * 16) * N_SH + panel * 64 + c * 4;
#pragma unroll
    for (int kb = 0; kb < 8; ++kb) {
      unsigned r[16];
#pragma unroll
      for (int j = 0; j < 16; ++j) r[j] = *(const unsigned*)(wp + (size_t)(kb * 64 + j) * N_SH);
#pragma unroll
      for (int q = 0; q < 4; ++q) {
        const unsigned t0 = perm(r[4 * q + 1], r[4 * q + 0], 0x05010400u);
        const unsigned t1 = perm(r[4 * q + 1], r[4 * q + 0], 0x07030602u);
        const unsigned t2 = perm(r[4 * q + 3], r[4 * q + 2], 0x05010400u);
        const unsigned t3 = perm(r[4 * q + 3], r[4 * q + 2], 0x07030602u);
        B[kb][0][q] = (int)perm(t2, t0, 0x05040100u);
        B[kb][1][q] = (int)perm(t2, t0, 0x07060302u);
        B[kb][2][q] = (int)perm(t3, t1, 0x05040100u);
        B[kb][3][q] = (int)perm(t3, t1, 0x07060302u);
      }
    }
  }
  const float scale = p.sx[0] * p.sw[0];
  const unsigned call = __builtin_amdgcn_readfirstlane(cprev) + 1u;
  if (lane == 0) __hip_atomic_store(myctr, call, __ATOMIC_RELAXED, __HIP_MEMORY_SCOPE_AGENT);
  unsigned* tmo = (unsigned*)(p.ws + TMO_OFF);
  const unsigned bufoff = (unsigned)(call % 3u) * (unsigned)BUF_BYTES;
  const __amdgpu_buffer_rsrc_t rsW = __builtin_amdgcn_make_buffer_rsrc(p.ws, 0, (int)DBG_OFF, 0x00020000);
  const __amdgpu_buffer_rsrc_t rsX = __builtin_amdgcn_make_buffer_rsrc((void*)p.x, 0, (int)SHARD, 0x00020000);
  const __amdgpu_buffer_rsrc_t rsO = __builtin_amdgcn_make_buffer_rsrc((void*)p.out, 0, NR * M_SH * N_SH * 4, 0x00020000);
  __syncthreads();

  const int nlocal = (grp >= 8 && grp < 16) ? 2 : 1;
  const int nremote = (grp < 8) ? 10 : 9;
  const int ntask = nlocal + nremote;
  for (int t = 0; t < ntask; ++t) {
    int src, rb;
    v4i acc[4];
    if (t < nlocal) {
      src = p.rank;
      rb = (t == 0) ? grp : 16 + grp;
      const int off = (rb * 16 + c) * K_DIM + wave * 512 + g * 16;
      v4i a[8];
#pragma unroll
      for (int kb = 0; kb < 8; ++kb)
        a[kb] = __builtin_bit_cast(v4i, __builtin_amdgcn_raw_buffer_load_b128(rsX, off + kb * 64, 0, 0));
      __builtin_amdgcn_sched_barrier(0);
#pragma unroll
      for (int cb = 0; cb < 4; ++cb) acc[cb] = (v4i){0, 0, 0, 0};
#pragma unroll
      for (int kb = 0; kb < 8; ++kb)
#pragma unroll
        for (int cb = 0; cb < 4; ++cb) acc[cb] = __builtin_amdgcn_mfma_i32_16x16x64_i8(a[kb], B[kb][cb], acc[cb], 0, 0, 0);
    } else {
      const int n = grp + NGROUP * (t - nlocal);
      rb = n / 7;
      src = (p.rank + 1 + (n - rb * 7)) & 7;
      const unsigned base = bufoff + (unsigned)src * (unsigned)SHARD + (unsigned)(rb * 65536 + wave * 8192);
      const int off = (int)base + lane * 16;
      const unsigned u0 = (unsigned)(rb * 16 + wave * 2) % (unsigned)SEND_W;
      const int poff = (lane >= 4) ? (int)TMO_OFF
                       : (lane & 2) ? (int)(FLAG_OFF + ((unsigned)src * 64u + u0 + (lane & 1)) * 128u)
                                    : (int)(base + 4092u + (lane & 1) * 4096u);
      const unsigned pref = (lane >= 2) ? call : 1u;
      const bool aggr = rb >= AGGR_RB;
      bool flags_ok = false;
      const unsigned long long t0 = __builtin_amdgcn_s_memrealtime();
      for (unsigned iter = 1;; ++iter) {
        const unsigned pv = __builtin_amdgcn_raw_buffer_load_b32(rsW, poff, 0, 17);
        bool go = true;
        unsigned bits = 0;
        if (!aggr) {
          bits = (unsigned)__builtin_amdgcn_ballot_w64(pv >= pref);
          flags_ok = (bits & 12u) == 12u;
          go = ((bits | (bits >> 2)) & 3u) == 3u;
        }
        if (go) {
          v4i a[8];
#pragma unroll
          for (int kb = 0; kb < 8; ++kb)
            a[kb] = __builtin_bit_cast(v4i, __builtin_amdgcn_raw_buffer_load_b128(rsW, off + kb * 1024, 0, A_AUX));
          __builtin_amdgcn_sched_barrier(0);
#pragma unroll
          for (int cb = 0; cb < 4; ++cb) acc[cb] = (v4i){0, 0, 0, 0};
          unsigned m = 0xffffffffu;
#pragma unroll
          for (int kb = 0; kb < 8; ++kb) {
            m = umin3(m, (unsigned)a[kb][0], (unsigned)a[kb][1]);
            m = umin3(m, (unsigned)a[kb][2], (unsigned)a[kb][3]);
            asm volatile("" : "+v"(m));
            const v4i av = a[kb] ^ XORK;
#pragma unroll
            for (int cb = 0; cb < 4; ++cb) acc[cb] = __builtin_amdgcn_mfma_i32_16x16x64_i8(av, B[kb][cb], acc[cb], 0, 0, 0);
          }
          if (flags_ok || __builtin_amdgcn_ballot_w64(m == 0u) == 0ull) break;
        }
        if (aggr) {
          bits = (unsigned)__builtin_amdgcn_ballot_w64(pv >= pref);
          flags_ok = (bits & 12u) == 12u;
        }
        if ((bits & 16u) || ((iter & 15u) == 0u && __builtin_amdgcn_s_memrealtime() - t0 > DEADLINE_TICKS)) {
          __hip_atomic_store(tmo, call, __ATOMIC_RELAXED, __HIP_MEMORY_SCOPE_AGENT);
          bad_flag = 1;
#pragma unroll
          for (int cb = 0; cb < 4; ++cb) acc[cb] = (v4i){0, 0, 0, 0};
          break;
        }
        __builtin_amdgcn_s_sleep(1);
      }
    }
    v4i* rw = &red[t & 1][wave][0];
#pragma unroll
    for (int i = 0; i < 4; ++i) rw[(4 * g + i) * 16 + c] = (v4i){acc[0][i], acc[1][i], acc[2][i], acc[3][i]};
    asm volatile("s_waitcnt lgkmcnt(0)" ::: "memory");
    __builtin_amdgcn_s_barrier();
    asm volatile("" ::: "memory");
    if (wave < 4) {
      v4i s = red[t & 1][0][wave * 64 + lane];
#pragma unroll
      for (int w2 = 1; w2 < NWAVE; ++w2) s += red[t & 1][w2][wave * 64 + lane];
      v4f o;
#pragma unroll
      for (int i = 0; i < 4; ++i) o[i] = silu_dq(s[i], scale);
      if (bad_flag) o = (v4f){__builtin_nanf(""), __builtin_nanf(""), __builtin_nanf(""), __builtin_nanf("")};
      const int row = src * M_SH + rb * 16 + wave * 4 + g;
      __builtin_amdgcn_raw_buffer_store_b128(__builtin_bit_cast(v4u, o), rsO, (row * N_SH + panel * 64 + c * 4) * 4, 0, O_AUX);
    }
  }
}

}

extern "C" void kernel_launch(int rank, int nranks, ncclComm_t comm, hipStream_t stream,
                              void* const* d_in, const int* in_sizes, int n_in,
                              void* d_out, int out_size,
                              const int* in_dtypes, int out_dtype,
                              void* const* d_out_all,
                              void* const* d_ws_all, size_t ws_bytes) {
  (void)comm; (void)in_sizes; (void)in_dtypes; (void)out_dtype; (void)d_out_all;
  if (nranks != NR || (unsigned)rank >= (unsigned)NR || n_in < 4 || ws_bytes < WS_NEED ||
      out_size != NR * M_SH * N_SH) {
    fprintf(stderr, "unexpected problem configuration\n");
    return;
  }
  Params p;
  for (int j = 0; j < 2 * (NR - 1); ++j) p.peer[j] = (char*)d_ws_all[(rank + 1 + j % (NR - 1)) % NR];
  p.ws = (char*)d_ws_all[rank];
  p.x = (const signed char*)d_in[0];
  p.w = (const signed char*)d_in[1];
  p.sx = (const float*)d_in[2];
  p.sw = (const float*)d_in[3];
  p.out = (float*)d_out;
  p.rank = rank;
  hipLaunchKernelGGL(ag_gemm_kernel, dim3(NWG), dim3(512), 0, stream, p);
  HIP_CHECK(hipGetLastError());
}
```

```cpp
#include <hip/hip_runtime.h>
#include <rccl/rccl.h>
#include <cstdio>
#define DIAG 1

#define HIP_CHECK(expr) do { hipError_t _e = (expr); if (_e != hipSuccess) { \
    fprintf(stderr, "HIP error %s at %s:%d\n", hipGetErrorString(_e), __FILE__, __LINE__); \
    return; } } while (0)

#define SEND_W 16
#define SEND_P 8
#define A_AUX 17
#define O_AUX 17
#define Z_AUX 17
#define AGGR_RB 31
#define Z_DELAY 300
#define Z_PACE 8

namespace {

typedef int v4i __attribute__((ext_vector_type(4)));
typedef unsigned v4u __attribute__((ext_vector_type(4)));
typedef float v4f __attribute__((ext_vector_type(4)));

constexpr int NR = 8;
constexpr int M_SH = 512, K_DIM = 4096, N_SH = 512;
constexpr size_t SHARD = (size_t)M_SH * K_DIM;
constexpr int UNITS = 512;
constexpr size_t BUF_BYTES = (size_t)NR * SHARD;
constexpr size_t STAGE_OFF = 0;
constexpr size_t FLAG_OFF = 3 * BUF_BYTES;
constexpr size_t CTR_OFF = FLAG_OFF + (size_t)NR * 64 * 128;
constexpr int NWG = 256, NWAVE = 8;
constexpr size_t TMO_OFF = CTR_OFF + (size_t)NWG * NWAVE * 4;
constexpr size_t DBG_OFF = TMO_OFF + 128;
constexpr size_t DBG_WORDS = (size_t)NWG * NWAVE * 16;
constexpr size_t WS_NEED = DBG_OFF + DBG_WORDS * 8;
constexpr unsigned long long DEADLINE_TICKS = 100000000ull;
constexpr int NGROUP = 24;
constexpr int XORK = (int)0x80808080u;

struct Params {
  char* peer[2 * (NR - 1)];
  char* ws;
  const signed char* x;
  const signed char* w;
  const float* sx;
  const float* sw;
  float* out;
  int rank;
};

#define WAITVM(n) asm volatile("s_waitcnt vmcnt(%0)" ::"n"(n) : "memory")

template <int IMM>
__device__ __forceinline__ void st_stage(unsigned voff, v4i d, char* sbase) {
  asm volatile("global_store_dwordx4 %0, %1, %2 offset:%3 sc0 sc1" ::"v"(voff), "v"(d), "s"(sbase), "n"(IMM) : "memory");
}
template <int IMM>
__device__ __forceinline__ void ld_x(v4i& d, unsigned voff, const signed char* sbase) {
  asm volatile("global_load_dwordx4 %0, %1, %2 offset:%3" : "+v"(d) : "v"(voff), "s"(sbase), "n"(IMM) : "memory");
}

template <int F, int WCNT>
__device__ __forceinline__ void send_frag(char* const (&dst)[NR - 1], unsigned vdst, v4i& d, unsigned vs, const signed char* x) {
  asm volatile("s_waitcnt vmcnt(%1)" : "+v"(d) : "n"(WCNT) : "memory");
  d ^= XORK;
#pragma unroll
  for (int j = 0; j < NR - 1; ++j) st_stage<F * 1024>(vdst, d, dst[j]);
  ld_x<F * 64>(d, vs, x);
}

template <int W, int P>
__device__ __forceinline__ void sender(const Params& p, int v, int lane, unsigned* myctr) {
  constexpr int UPI = P / 4;
  constexpr int NU = UNITS / W;
  constexpr int NIT = NU / UPI;
  constexpr unsigned USTEP = (unsigned)W * 4096u;
  constexpr int WCNT = 8 * (P - 1) < 54 ? 8 * (P - 1) : 54;

  unsigned vsrc = (unsigned)((v >> 4) * 65536 + (v & 15) * 256 + (lane & 15) * 4096 + (lane >> 4) * 16);
  v4i buf[P];
#pragma unroll
  for (int s = 0; s < P; ++s) buf[s] = (v4i){0, 0, 0, 0};
  const unsigned cprev = __hip_atomic_load(myctr, __ATOMIC_RELAXED, __HIP_MEMORY_SCOPE_AGENT);
  {
    unsigned vs = vsrc;
#pragma unroll
    for (int uu = 0; uu < UPI; ++uu) {
      ld_x<0>(buf[uu * 4 + 0], vs, p.x);
      ld_x<64>(buf[uu * 4 + 1], vs, p.x);
      ld_x<128>(buf[uu * 4 + 2], vs, p.x);
      ld_x<192>(buf[uu * 4 + 3], vs, p.x);
      vs += USTEP;
    }
  }
  char* dst[NR - 1];
  {
    char* const* pp = p.peer + v % (NR - 1);
#pragma unroll
    for (int j = 0; j < NR - 1; ++j) dst[j] = pp[j];
    asm volatile("" : "+s"(dst[0]), "+s"(dst[1]), "+s"(dst[2]), "+s"(dst[3]), "+s"(dst[4]), "+s"(dst[5]), "+s"(dst[6]));
  }
  const unsigned call = __builtin_amdgcn_readfirstlane(cprev) + 1u;
  unsigned vdst = (unsigned)STAGE_OFF + (call % 3u) * (unsigned)BUF_BYTES + (unsigned)p.rank * (unsigned)SHARD +
                  (unsigned)(v * 4096 + lane * 16);
  WAITVM(0);

  for (int it = 0; it < NIT; ++it) {
    const bool more = (it + 1 < NIT);
#pragma unroll
    for (int uu = 0; uu < UPI; ++uu) {
      const unsigned vs = vsrc + (more ? (unsigned)(UPI + uu) * USTEP : 0u);
      const unsigned vd = vdst + (unsigned)uu * USTEP;
      send_frag<0, WCNT>(dst, vd, buf[uu * 4 + 0], vs, p.x);
      send_frag<1, WCNT>(dst, vd, buf[uu * 4 + 1], vs, p.x);
      send_frag<2, WCNT>(dst, vd, buf[uu * 4 + 2], vs, p.x);
      send_frag<3, WCNT>(dst, vd, buf[uu * 4 + 3], vs, p.x);
    }
    vsrc += (unsigned)UPI * USTEP;
    vdst += (unsigned)UPI * USTEP;
  }
  WAITVM(0);
#pragma unroll
  for (int s = 0; s < P; ++s) asm volatile("" : "+v"(buf[s])::"memory");
  if (lane == 0) {
    const size_t foff = FLAG_OFF + ((size_t)p.rank * 64 + v) * 128;
#pragma unroll
    for (int j = 0; j < NR - 1; ++j)
      __hip_atomic_store((unsigned*)(dst[j] + foff), call, __ATOMIC_RELAXED, __HIP_MEMORY_SCOPE_SYSTEM);
    __hip_atomic_store(myctr, call, __ATOMIC_RELAXED, __HIP_MEMORY_SCOPE_AGENT);
  }
}

__device__ __forceinline__ unsigned perm(unsigned hi, unsigned lo, unsigned sel) {
  return __builtin_amdgcn_perm(hi, lo, sel);
}

__device__ __forceinline__ float silu_dq(int acc, float scale) {
  const float y = (float)acc * scale;
  const float yc = fminf(fmaxf(y, -60.0f), 60.0f);
  return y * __builtin_amdgcn_rcpf(1.0f + __expf(-yc));
}

__device__ __forceinline__ unsigned umin3(unsigned a, unsigned b, unsigned c) { return min(min(a, b), c); }

__global__ void __launch_bounds__(512, 2) ag_gemm_kernel(Params p) {
  __shared__ v4i red[2][NWAVE][256];
  __shared__ int bad_flag;
  const int b = blockIdx.x, tid = threadIdx.x, lane = tid & 63;
  const int wave = __builtin_amdgcn_readfirstlane(tid >> 6);
  const int xcd = b & 7, idx = b >> 3;
  unsigned* myctr = (unsigned*)(p.ws + CTR_OFF) + b * NWAVE + wave;

  if (idx < 8) {
    if (b < SEND_W) {
      if (wave == 0) {
        sender<SEND_W, SEND_P>(p, b, lane, myctr);
      }
    } else {
      const unsigned long long tz = __builtin_amdgcn_s_memrealtime();
      const unsigned cprev = __hip_atomic_load(myctr, __ATOMIC_RELAXED, __HIP_MEMORY_SCOPE_AGENT);
      const unsigned call = __builtin_amdgcn_readfirstlane(cprev) + 1u;
      const __amdgpu_buffer_rsrc_t rsZ =
          __builtin_amdgcn_make_buffer_rsrc(p.ws + STAGE_OFF + (size_t)((call + 2u) % 3u) * BUF_BYTES, 0, (int)BUF_BYTES, 0x00020000);
      if (lane == 0) __hip_atomic_store(myctr, call, __ATOMIC_RELAXED, __HIP_MEMORY_SCOPE_AGENT);
      if (b == 63 && wave == 7) {
        unsigned long long* prev = (unsigned long long*)(p.ws + DBG_OFF);
        const unsigned long long pe = *prev;
        const int sj = lane & 7, src = (p.rank + 1 + sj) & 7, which = (lane >> 3) & 3;
        const unsigned uo[4] = {4092u, 128u * 4096u + 4092u, 256u * 4096u + 4092u, 511u * 4096u + 4092u};
        const unsigned* pa = (const unsigned*)(p.ws + STAGE_OFF + (size_t)(call % 3u) * BUF_BYTES + (size_t)src * SHARD +
                                               (which == 0 ? uo[0] : which == 1 ? uo[1] : which == 2 ? uo[2] : uo[3]));
        unsigned long long ta = 0;
        bool act = lane < 32 && sj < 7;
        for (;;) {
          const unsigned v = __hip_atomic_load(pa, __ATOMIC_RELAXED, __HIP_MEMORY_SCOPE_SYSTEM);
          const unsigned long long now = __builtin_amdgcn_s_memrealtime();
          if (act && v != 0u) { ta = now; act = false; }
          if (__builtin_amdgcn_ballot_w64(act) == 0ull || now - tz > DEADLINE_TICKS) break;
          __builtin_amdgcn_s_sleep(2);
        }
        const unsigned long long te = __builtin_amdgcn_s_memrealtime();
        int rel = (int)(ta - tz);
        int a[28];
#pragma unroll
        for (int w4 = 0; w4 < 4; ++w4)
#pragma unroll
          for (int j = 0; j < 7; ++j) a[w4 * 7 + j] = __builtin_amdgcn_readlane(rel, w4 * 8 + j);
        if (lane == 0) {
          if (pe == 0ull || tz - pe > 20000ull)
            printf("DIAG r%d c%u idle %lld | f %d %d %d %d %d %d %d | q1 %d %d %d %d %d %d %d | h %d %d %d %d %d %d %d | l %d %d %d %d %d %d %d\n", p.rank, call,
                   (long long)(pe ? tz - pe : 0ull), a[0], a[1], a[2], a[3], a[4], a[5], a[6], a[7], a[8], a[9], a[10], a[11], a[12], a[13],
                   a[14], a[15], a[16], a[17], a[18], a[19], a[20], a[21], a[22], a[23], a[24], a[25], a[26], a[27]);
          *prev = __builtin_amdgcn_s_memrealtime();
        }
        (void)te;
        return;
      }
      constexpr int NZ = (64 - SEND_W) * NWAVE - 1;
      while (__builtin_amdgcn_s_memrealtime() - tz < Z_DELAY) __builtin_amdgcn_s_sleep(8);
      for (int ch = (b - SEND_W) * NWAVE + wave; ch < 7 * 2048; ch += NZ) {
        const int off = ((p.rank + 1 + (ch >> 11)) & 7) * (int)SHARD + (ch & 2047) * 1024 + lane * 16;
        __builtin_amdgcn_raw_buffer_store_b128((v4u){0u, 0u, 0u, 0u}, rsZ, off, 0, Z_AUX);
        __builtin_amdgcn_s_sleep(Z_PACE);
      }
    }
    return;
  }

  const int cidx = idx - 8;
  const int grp = (cidx >> 3) * 8 + xcd;
  const int panel = cidx & 7;
  const int c = lane & 15, g = lane >> 4;
  const unsigned cprev = __hip_atomic_load(myctr, __ATOMIC_RELAXED, __HIP_MEMORY_SCOPE_AGENT);
  if (tid == 0) bad_flag = 0;

  v4i B[8][4];
  {
    const signed char* wp = p.w + (size_t)(wave * 512 + g * 16) * N_SH + panel * 64 + c * 4;
#pragma unroll
    for (int kb = 0; kb < 8; ++kb) {
      unsigned r[16];
#pragma unroll
      for (int j = 0; j < 16; ++j) r[j] = *(const unsigned*)(wp + (size_t)(kb * 64 + j) * N_SH);
#pragma unroll
      for (int q = 0; q < 4; ++q) {
        const unsigned t0 = perm(r[4 * q + 1], r[4 * q + 0], 0x05010400u);
        const unsigned t1 = perm(r[4 * q + 1], r[4 * q + 0], 0x07030602u);
        const unsigned t2 = perm(r[4 * q + 3], r[4 * q + 2], 0x05010400u);
        const unsigned t3 = perm(r[4 * q + 3], r[4 * q + 2], 0x07030602u);
        B[kb][0][q] = (int)perm(t2, t0, 0x05040100u);
        B[kb][1][q] = (int)perm(t2, t0, 0x07060302u);
        B[kb][2][q] = (int)perm(t3, t1, 0x05040100u);
        B[kb][3][q] = (int)perm(t3, t1, 0x07060302u);
      }
    }
  }
  const float scale = p.sx[0] * p.sw[0];
  const unsigned call = __builtin_amdgcn_readfirstlane(cprev) + 1u;
  if (lane == 0) __hip_atomic_store(myctr, call, __ATOMIC_RELAXED, __HIP_MEMORY_SCOPE_AGENT);
  unsigned* tmo = (unsigned*)(p.ws + TMO_OFF);
  const unsigned bufoff = (unsigned)(call % 3u) * (unsigned)BUF_BYTES;
  const __amdgpu_buffer_rsrc_t rsW = __builtin_amdgcn_make_buffer_rsrc(p.ws, 0, (int)DBG_OFF, 0x00020000);
  const __amdgpu_buffer_rsrc_t rsX = __builtin_amdgcn_make_buffer_rsrc((void*)p.x, 0, (int)SHARD, 0x00020000);
  const __amdgpu_buffer_rsrc_t rsO = __builtin_amdgcn_make_buffer_rsrc((void*)p.out, 0, NR * M_SH * N_SH * 4, 0x00020000);
  __syncthreads();

  const int nlocal = (grp >= 8 && grp < 16) ? 2 : 1;
  const int nremote = (grp < 8) ? 10 : 9;
  const int ntask = nlocal + nremote;
  for (int t = 0; t < ntask; ++t) {
    int src, rb;
    v4i acc[4];
    if (t < nlocal) {
      src = p.rank;
      rb = (t == 0) ? grp : 16 + grp;
      const int off = (rb * 16 + c) * K_DIM + wave * 512 + g * 16;
      v4i a[8];
#pragma unroll
      for (int kb = 0; kb < 8; ++kb)
        a[kb] = __builtin_bit_cast(v4i, __builtin_amdgcn_raw_buffer_load_b128(rsX, off + kb * 64, 0, 0));
      __builtin_amdgcn_sched_barrier(0);
#pragma unroll
      for (int cb = 0; cb < 4; ++cb) acc[cb] = (v4i){0, 0, 0, 0};
#pragma unroll
      for (int kb = 0; kb < 8; ++kb)
#pragma unroll
        for (int cb = 0; cb < 4; ++cb) acc[cb] = __builtin_amdgcn_mfma_i32_16x16x64_i8(a[kb], B[kb][cb], acc[cb], 0, 0, 0);
    } else {
      const int n = grp + NGROUP * (t - nlocal);
      rb = n / 7;
      src = (p.rank + 1 + (n - rb * 7)) & 7;
      const unsigned base = bufoff + (unsigned)src * (unsigned)SHARD + (unsigned)(rb * 65536 + wave * 8192);
      const int off = (int)base + lane * 16;
      const unsigned u0 = (unsigned)(rb * 16 + wave * 2) % (unsigned)SEND_W;
      const int poff = (lane >= 4) ? (int)TMO_OFF
                       : (lane & 2) ? (int)(FLAG_OFF + ((unsigned)src * 64u + u0 + (lane & 1)) * 128u)
                                    : (int)(base + 4092u + (lane & 1) * 4096u);
      const unsigned pref = (lane >= 2) ? call : 1u;
      const bool aggr = rb >= AGGR_RB;
      bool flags_ok = false;
      const unsigned long long t0 = __builtin_amdgcn_s_memrealtime();
      for (unsigned iter = 1;; ++iter) {
        const unsigned pv = __builtin_amdgcn_raw_buffer_load_b32(rsW, poff, 0, 17);
        bool go = true;
        unsigned bits = 0;
        if (!aggr) {
          bits = (unsigned)__builtin_amdgcn_ballot_w64(pv >= pref);
          flags_ok = (bits & 12u) == 12u;
          go = ((bits | (bits >> 2)) & 3u) == 3u;
        }
        if (go) {
          v4i a[8];
#pragma unroll
          for (int kb = 0; kb < 8; ++kb)
            a[kb] = __builtin_bit_cast(v4i, __builtin_amdgcn_raw_buffer_load_b128(rsW, off + kb * 1024, 0, A_AUX));
          __builtin_amdgcn_sched_barrier(0);
#pragma unroll
          for (int cb = 0; cb < 4; ++cb) acc[cb] = (v4i){0, 0, 0, 0};
          unsigned m = 0xffffffffu;
#pragma unroll
          for (int kb = 0; kb < 8; ++kb) {
            m = umin3(m, (unsigned)a[kb][0], (unsigned)a[kb][1]);
            m = umin3(m, (unsigned)a[kb][2], (unsigned)a[kb][3]);
            asm volatile("" : "+v"(m));
            const v4i av = a[kb] ^ XORK;
#pragma unroll
            for (int cb = 0; cb < 4; ++cb) acc[cb] = __builtin_amdgcn_mfma_i32_16x16x64_i8(av, B[kb][cb], acc[cb], 0, 0, 0);
          }
          if (flags_ok || __builtin_amdgcn_ballot_w64(m == 0u) == 0ull) break;
        }
        if (aggr) {
          bits = (unsigned)__builtin_amdgcn_ballot_w64(pv >= pref);
          flags_ok = (bits & 12u) == 12u;
        }
        if ((bits & 16u) || ((iter & 15u) == 0u && __builtin_amdgcn_s_memrealtime() - t0 > DEADLINE_TICKS)) {
          __hip_atomic_store(tmo, call, __ATOMIC_RELAXED, __HIP_MEMORY_SCOPE_AGENT);
          bad_flag = 1;
#pragma unroll
          for (int cb = 0; cb < 4; ++cb) acc[cb] = (v4i){0, 0, 0, 0};
          break;
        }
        __builtin_amdgcn_s_sleep(1);
      }
    }
    v4i* rw = &red[t & 1][wave][0];
#pragma unroll
    for (int i = 0; i < 4; ++i) rw[(4 * g + i) * 16 + c] = (v4i){acc[0][i], acc[1][i], acc[2][i], acc[3][i]};
    asm volatile("s_waitcnt lgkmcnt(0)" ::: "memory");
    __builtin_amdgcn_s_barrier();
    asm volatile("" ::: "memory");
    if (wave < 4) {
      v4i s = red[t & 1][0][wave * 64 + lane];
#pragma unroll
      for (int w2 = 1; w2 < NWAVE; ++w2) s += red[t & 1][w2][wave * 64 + lane];
      v4f o;
#pragma unroll
      for (int i = 0; i < 4; ++i) o[i] = silu_dq(s[i], scale);
      if (bad_flag) o = (v4f){__builtin_nanf(""), __builtin_nanf(""), __builtin_nanf(""), __builtin_nanf("")};
      const int row = src * M_SH + rb * 16 + wave * 4 + g;
      __builtin_amdgcn_raw_buffer_store_b128(__builtin_bit_cast(v4u, o), rsO, (row * N_SH + panel * 64 + c * 4) * 4, 0, O_AUX);
    }
  }
}

}

extern "C" void kernel_launch(int rank, int nranks, ncclComm_t comm, hipStream_t stream,
                              void* const* d_in, const int* in_sizes, int n_in,
                              void* d_out, int out_size,
                              const int* in_dtypes, int out_dtype,
                              void* const* d_out_all,
                              void* const* d_ws_all, size_t ws_bytes) {
  (void)comm; (void)in_sizes; (void)in_dtypes; (void)out_dtype; (void)d_out_all;
  if (nranks != NR || (unsigned)rank >= (unsigned)NR || n_in < 4 || ws_bytes < WS_NEED ||
      out_size != NR * M_SH * N_SH) {
    fprintf(stderr, "unexpected problem configuration\n");
    return;
  }
  Params p;
  for (int j = 0; j < 2 * (NR - 1); ++j) p.peer[j] = (char*)d_ws_all[(rank + 1 + j % (NR - 1)) % NR];
  p.ws = (char*)d_ws_all[rank];
  p.x = (const signed char*)d_in[0];
  p.w = (const signed char*)d_in[1];
  p.sx = (const float*)d_in[2];
  p.sw = (const float*)d_in[3];
  p.out = (float*)d_out;
  p.rank = rank;
  hipLaunchKernelGGL(ag_gemm_kernel, dim3(NWG), dim3(512), 0, stream, p);
  HIP_CHECK(hipGetLastError());
}
```

```cpp
#include <hip/hip_runtime.h>
#include <rccl/rccl.h>
#include <cstdio>

#define HIP_CHECK(expr) do { hipError_t _e = (expr); if (_e != hipSuccess) { \
    fprintf(stderr, "HIP error %s at %s:%d\n", hipGetErrorString(_e), __FILE__, __LINE__); \
    return; } } while (0)

#define SEND_W 16
#define A_AUX 17
#define O_AUX 17
#define Z_AUX 17
#define AGGR_RB 31
#define Z_DELAY 300
#define Z_PACE 8

namespace {

typedef int v4i __attribute__((ext_vector_type(4)));
typedef unsigned v4u __attribute__((ext_vector_type(4)));
typedef float v4f __attribute__((ext_vector_type(4)));

constexpr int NR = 8;
constexpr int M_SH = 512, K_DIM = 4096, N_SH = 512;
constexpr size_t SHARD = (size_t)M_SH * K_DIM;
constexpr int UNITS = 512;
constexpr size_t BUF_BYTES = (size_t)NR * SHARD;
constexpr size_t STAGE_OFF = 0;
constexpr size_t FLAG_OFF = 3 * BUF_BYTES;
constexpr size_t TKT_OFF = FLAG_OFF + (size_t)NR * 128;
constexpr size_t CTR_OFF = TKT_OFF + 128;
constexpr int NWG = 256, NWAVE = 8;
constexpr size_t TMO_OFF = CTR_OFF + (size_t)NWG * NWAVE * 4;
constexpr size_t DBG_OFF = TMO_OFF + 128;
constexpr size_t DBG_WORDS = (size_t)NWG * NWAVE * 16;
constexpr size_t WS_NEED = DBG_OFF + DBG_WORDS * 8;
constexpr unsigned long long DEADLINE_TICKS = 100000000ull;
constexpr int NGROUP = 24;
constexpr int XORK = (int)0x80808080u;

struct Params {
  char* peer[2 * (NR - 1)];
  char* ws;
  const signed char* x;
  const signed char* w;
  const float* sx;
  const float* sw;
  float* out;
  int rank;
};

#define WAITVM(n) asm volatile("s_waitcnt vmcnt(%0)" ::"n"(n) : "memory")

__device__ __forceinline__ unsigned unit_src(unsigned u) { return (u >> 4) * 65536u + (u & 15u) * 256u; }

#define S_ST(D, R, OD) "global_store_dwordx4 v67, " R ", %[" D "] offset:" OD " sc0 sc1\n\t"
#define S_XOR(R) "v_xor_b32 " R ", %[xk], " R "\n\t"
#define S_FRAG(R0, R1, R2, R3, R, OD, OS, PRE)                                       \
  S_XOR(R0) S_XOR(R1) S_XOR(R2) S_XOR(R3)                                            \
  S_ST("d0", R, OD) S_ST("d1", R, OD) S_ST("d2", R, OD) S_ST("d3", R, OD)            \
  S_ST("d4", R, OD) S_ST("d5", R, OD) S_ST("d6", R, OD) PRE                          \
  "global_load_dwordx4 " R ", v66, %[x] offset:" OS "\n\t"
#define S_W54 "s_waitcnt vmcnt(54)\n\t"
#define S_HEAD(TK, U)                                                                \
  "s_waitcnt vmcnt(53)\n\t"                                                          \
  "v_readfirstlane_b32 s43, " TK "\n\t"                                              \
  "s_mov_b64 exec, 1\n\t"                                                            \
  "global_atomic_add " TK ", %[vtk], %[one], %[ws] sc0\n\t"                          \
  "s_mov_b64 exec, -1\n\t"                                                           \
  "v_lshl_add_u32 v67, " U ", 12, v68\n\t"                                           \
  "s_sub_u32 " U ", s43, s42\n\t"                                                    \
  "s_min_u32 s43, " U ", 511\n\t"                                                    \
  "s_lshl_b32 s44, s43, 12\n\t"                                                      \
  "s_lshl_b32 s43, s43, 8\n\t"                                                       \
  "s_and_b32 s44, s44, 0x1f0000\n\t"                                                 \
  "s_and_b32 s43, s43, 0xf00\n\t"                                                    \
  "s_or_b32 s43, s43, s44\n\t"
#define S_VS "v_or_b32 v66, s43, %[lsrc]\n\t"

template <int W>
__device__ __forceinline__ void sender(const Params& p, int v, int lane) {
  static_assert(UNITS == 512 && BUF_BYTES == (1u << 24) && STAGE_OFF == 0, "asm constants");
  const unsigned lsrc = (unsigned)((lane & 15) * 4096 + (lane >> 4) * 16);
  const unsigned vsA = lsrc + unit_src((unsigned)v), vsB = lsrc + unit_src((unsigned)(v + W));
  const unsigned ldst = (unsigned)p.rank * (unsigned)SHARD + (unsigned)(lane * 16);
  const unsigned vctr = (unsigned)CTR_OFF + (unsigned)(v * NWAVE * 4);
  char* const* pp = p.peer + v % (NR - 1);
  char* const d0 = pp[0]; char* const d1 = pp[1]; char* const d2 = pp[2]; char* const d3 = pp[3];
  char* const d4 = pp[4]; char* const d5 = pp[5]; char* const d6 = pp[6];
  unsigned call;
  asm volatile(
      "global_load_dword v66, %[vctr], %[ws] sc1\n\t"
      "global_load_dwordx4 v[32:35], %[vsA], %[x] offset:0\n\t"
      "global_load_dwordx4 v[36:39], %[vsA], %[x] offset:64\n\t"
      "global_load_dwordx4 v[40:43], %[vsA], %[x] offset:128\n\t"
      "global_load_dwordx4 v[44:47], %[vsA], %[x] offset:192\n\t"
      "global_load_dwordx4 v[48:51], %[vsB], %[x] offset:0\n\t"
      "global_load_dwordx4 v[52:55], %[vsB], %[x] offset:64\n\t"
      "global_load_dwordx4 v[56:59], %[vsB], %[x] offset:128\n\t"
      "global_load_dwordx4 v[60:63], %[vsB], %[x] offset:192\n\t"
      "s_mov_b32 s40, %[v]\n\t"
      "s_add_u32 s41, %[v], %[w1]\n\t"
      "s_waitcnt vmcnt(8)\n\t"
      "v_readfirstlane_b32 s43, v66\n\t"
      "s_add_u32 %[call], s43, 1\n\t"
      "s_lshl_b32 s42, s43, 9\n\t"
      "s_sub_u32 s42, s42, %[w4]\n\t"
      "s_mul_hi_u32 s43, %[call], 0xaaaaaaab\n\t"
      "s_lshr_b32 s43, s43, 1\n\t"
      "s_mul_i32 s43, s43, 3\n\t"
      "s_sub_u32 s43, %[call], s43\n\t"
      "s_lshl_b32 s43, s43, 24\n\t"
      "v_add_u32 v68, s43, %[ldst]\n\t"
      "s_add_u32 s43, s42, s40\n\t"
      "s_add_u32 s44, s43, %[w3]\n\t"
      "s_add_u32 s43, s43, %[w2]\n\t"
      "v_mov_b32 v64, s43\n\t"
      "v_mov_b32 v65, s44\n\t"
      "s_waitcnt vmcnt(0)\n"
      ".Lsend_loop_%=:\n\t"
      S_HEAD("v64", "s40")
      S_FRAG("v32", "v33", "v34", "v35", "v[32:35]", "0", "0", S_VS) S_W54
      S_FRAG("v36", "v37", "v38", "v39", "v[36:39]", "1024", "64", "") S_W54
      S_FRAG("v40", "v41", "v42", "v43", "v[40:43]", "2048", "128", "") S_W54
      S_FRAG("v44", "v45", "v46", "v47", "v[44:47]", "3072", "192", "")
      "s_cmp_gt_u32 s41, 511\n\t"
      "s_cbranch_scc1 .Lsend_end_%=\n\t"
      S_HEAD("v65", "s41")
      S_FRAG("v48", "v49", "v50", "v51", "v[48:51]", "0", "0", S_VS) S_W54
      S_FRAG("v52", "v53", "v54", "v55", "v[52:55]", "1024", "64", "") S_W54
      S_FRAG("v56", "v57", "v58", "v59", "v[56:59]", "2048", "128", "") S_W54
      S_FRAG("v60", "v61", "v62", "v63", "v[60:63]", "3072", "192", "")
      "s_cmp_le_u32 s40, 511\n\t"
      "s_cbranch_scc1 .Lsend_loop_%=\n"
      ".Lsend_end_%=:\n\t"
      "s_waitcnt vmcnt(0)"
      : [call] "=&s"(call)
      : [d0] "s"(d0), [d1] "s"(d1), [d2] "s"(d2), [d3] "s"(d3), [d4] "s"(d4), [d5] "s"(d5), [d6] "s"(d6),
        [x] "s"(p.x), [ws] "s"(p.ws), [v] "s"(v), [xk] "s"(XORK), [vctr] "v"(vctr), [vtk] "v"((unsigned)TKT_OFF),
        [one] "v"(1u), [lsrc] "v"(lsrc), [ldst] "v"(ldst), [vsA] "v"(vsA), [vsB] "v"(vsB),
        [w1] "n"(W), [w2] "n"(2 * W), [w3] "n"(3 * W), [w4] "n"(4 * W)
      : "memory", "scc", "s40", "s41", "s42", "s43", "s44",
        "v32", "v33", "v34", "v35", "v36", "v37", "v38", "v39", "v40", "v41", "v42", "v43", "v44", "v45", "v46", "v47",
        "v48", "v49", "v50", "v51", "v52", "v53", "v54", "v55", "v56", "v57", "v58", "v59", "v60", "v61", "v62", "v63",
        "v64", "v65", "v66", "v67", "v68");
  if (lane == 0) {
    const size_t foff = FLAG_OFF + (size_t)p.rank * 128 + (size_t)v * 4;
    char* const dd[NR - 1] = {d0, d1, d2, d3, d4, d5, d6};
#pragma unroll
    for (int j = 0; j < NR - 1; ++j)
      __hip_atomic_store((unsigned*)(dd[j] + foff), call, __ATOMIC_RELAXED, __HIP_MEMORY_SCOPE_SYSTEM);
    __hip_atomic_store((unsigned*)(p.ws + vctr), call, __ATOMIC_RELAXED, __HIP_MEMORY_SCOPE_AGENT);
  }
}

__device__ __forceinline__ unsigned perm(unsigned hi, unsigned lo, unsigned sel) {
  return __builtin_amdgcn_perm(hi, lo, sel);
}

__device__ __forceinline__ float silu_dq(int acc, float scale) {
  const float y = (float)acc * scale;
  const float yc = fminf(fmaxf(y, -60.0f), 60.0f);
  return y * __builtin_amdgcn_rcpf(1.0f + __expf(-yc));
}

__device__ __forceinline__ unsigned umin3(unsigned a, unsigned b, unsigned c) { return min(min(a, b), c); }

__global__ void __launch_bounds__(512, 2) ag_gemm_kernel(Params p) {
  __shared__ v4i red[2][NWAVE][256];
  __shared__ int bad_flag;
  const int b = blockIdx.x, tid = threadIdx.x, lane = tid & 63;
  const int wave = __builtin_amdgcn_readfirstlane(tid >> 6);
  const int xcd = b & 7, idx = b >> 3;
  unsigned* myctr = (unsigned*)(p.ws + CTR_OFF) + b * NWAVE + wave;

  if (idx < 8) {
    if (b < SEND_W) {
      if (wave == 0) {
        sender<SEND_W>(p, b, lane);
      }
    } else {
      const unsigned long long tz = __builtin_amdgcn_s_memrealtime();
      const unsigned cprev = __hip_atomic_load(myctr, __ATOMIC_RELAXED, __HIP_MEMORY_SCOPE_AGENT);
      const unsigned call = __builtin_amdgcn_readfirstlane(cprev) + 1u;
      const __amdgpu_buffer_rsrc_t rsZ =
          __builtin_amdgcn_make_buffer_rsrc(p.ws + STAGE_OFF + (size_t)((call + 2u) % 3u) * BUF_BYTES, 0, (int)BUF_BYTES, 0x00020000);
      if (lane == 0) __hip_atomic_store(myctr, call, __ATOMIC_RELAXED, __HIP_MEMORY_SCOPE_AGENT);
      constexpr int NZ = (64 - SEND_W) * NWAVE;
      while (__builtin_amdgcn_s_memrealtime() - tz < Z_DELAY) __builtin_amdgcn_s_sleep(8);
      for (int ch = (b - SEND_W) * NWAVE + wave; ch < 7 * 2048; ch += NZ) {
        const int off = ((p.rank + 1 + (ch >> 11)) & 7) * (int)SHARD + (ch & 2047) * 1024 + lane * 16;
        __builtin_amdgcn_raw_buffer_store_b128((v4u){0u, 0u, 0u, 0u}, rsZ, off, 0, Z_AUX);
        __builtin_amdgcn_s_sleep(Z_PACE);
      }
    }
    return;
  }

  const int cidx = idx - 8;
  const int grp = (cidx >> 3) * 8 + xcd;
  const int panel = cidx & 7;
  const int c = lane & 15, g = lane >> 4;
  const unsigned cprev = __hip_atomic_load(myctr, __ATOMIC_RELAXED, __HIP_MEMORY_SCOPE_AGENT);
  if (tid == 0) bad_flag = 0;

  v4i B[8][4];
  {
    const signed char* wp = p.w + (size_t)(wave * 512 + g * 16) * N_SH + panel * 64 + c * 4;
#pragma unroll
    for (int kb = 0; kb < 8; ++kb) {
      unsigned r[16];
#pragma unroll
      for (int j = 0; j < 16; ++j) r[j] = *(const unsigned*)(wp + (size_t)(kb * 64 + j) * N_SH);
#pragma unroll
      for (int q = 0; q < 4; ++q) {
        const unsigned t0 = perm(r[4 * q + 1], r[4 * q + 0], 0x05010400u);
        const unsigned t1 = perm(r[4 * q + 1], r[4 * q + 0], 0x07030602u);
        const unsigned t2 = perm(r[4 * q + 3], r[4 * q + 2], 0x05010400u);
        const unsigned t3 = perm(r[4 * q + 3], r[4 * q + 2], 0x07030602u);
        B[kb][0][q] = (int)perm(t2, t0, 0x05040100u);
        B[kb][1][q] = (int)perm(t2, t0, 0x07060302u);
        B[kb][2][q] = (int)perm(t3, t1, 0x05040100u);
        B[kb][3][q] = (int)perm(t3, t1, 0x07060302u);
      }
    }
  }
  const float scale = p.sx[0] * p.sw[0];
  const unsigned call = __builtin_amdgcn_readfirstlane(cprev) + 1u;
  if (lane == 0) __hip_atomic_store(myctr, call, __ATOMIC_RELAXED, __HIP_MEMORY_SCOPE_AGENT);
  unsigned* tmo = (unsigned*)(p.ws + TMO_OFF);
  const unsigned bufoff = (unsigned)(call % 3u) * (unsigned)BUF_BYTES;
  const __amdgpu_buffer_rsrc_t rsW = __builtin_amdgcn_make_buffer_rsrc(p.ws, 0, (int)DBG_OFF, 0x00020000);
  const __amdgpu_buffer_rsrc_t rsX = __builtin_amdgcn_make_buffer_rsrc((void*)p.x, 0, (int)SHARD, 0x00020000);
  const __amdgpu_buffer_rsrc_t rsO = __builtin_amdgcn_make_buffer_rsrc((void*)p.out, 0, NR * M_SH * N_SH * 4, 0x00020000);
  __syncthreads();

  const int nlocal = (grp >= 8 && grp < 16) ? 2 : 1;
  const int nremote = (grp < 8) ? 10 : 9;
  const int ntask = nlocal + nremote;
  for (int t = 0; t < ntask; ++t) {
    int src, rb;
    v4i acc[4];
    if (t < nlocal) {
      src = p.rank;
      rb = (t == 0) ? grp : 16 + grp;
      const int off = (rb * 16 + c) * K_DIM + wave * 512 + g * 16;
      v4i a[8];
#pragma unroll
      for (int kb = 0; kb < 8; ++kb)
        a[kb] = __builtin_bit_cast(v4i, __builtin_amdgcn_raw_buffer_load_b128(rsX, off + kb * 64, 0, 0));
      __builtin_amdgcn_sched_barrier(0);
#pragma unroll
      for (int cb = 0; cb < 4; ++cb) acc[cb] = (v4i){0, 0, 0, 0};
#pragma unroll
      for (int kb = 0; kb < 8; ++kb)
#pragma unroll
        for (int cb = 0; cb < 4; ++cb) acc[cb] = __builtin_amdgcn_mfma_i32_16x16x64_i8(a[kb], B[kb][cb], acc[cb], 0, 0, 0);
    } else {
      const int n = grp + NGROUP * (t - nlocal);
      rb = n / 7;
      src = (p.rank + 1 + (n - rb * 7)) & 7;
      const unsigned base = bufoff + (unsigned)src * (unsigned)SHARD + (unsigned)(rb * 65536 + wave * 8192);
      const int off = (int)base + lane * 16;
      const int poff = (lane >= 32) ? (int)(FLAG_OFF + (unsigned)src * 128u + (unsigned)(lane & (SEND_W - 1)) * 4u)
                       : (lane >= 2) ? (int)TMO_OFF
                                     : (int)(base + 4092u + (lane & 1) * 4096u);
      const unsigned pref = (lane >= 2) ? call : 1u;
      const bool aggr = rb >= AGGR_RB;
      bool flags_ok = false;
      const unsigned long long t0 = __builtin_amdgcn_s_memrealtime();
      for (unsigned iter = 1;; ++iter) {
        const unsigned pv = __builtin_amdgcn_raw_buffer_load_b32(rsW, poff, 0, 17);
        bool go = true;
        unsigned long long bits = 0;
        if (!aggr) {
          bits = __builtin_amdgcn_ballot_w64(pv >= pref);
          flags_ok = (unsigned)(bits >> 32) == 0xffffffffu;
          go = flags_ok || ((unsigned)bits & 3u) == 3u;
        }
        if (go) {
          v4i a[8];
#pragma unroll
          for (int kb = 0; kb < 8; ++kb)
            a[kb] = __builtin_bit_cast(v4i, __builtin_amdgcn_raw_buffer_load_b128(rsW, off + kb * 1024, 0, A_AUX));
          __builtin_amdgcn_sched_barrier(0);
#pragma unroll
          for (int cb = 0; cb < 4; ++cb) acc[cb] = (v4i){0, 0, 0, 0};
          unsigned m = 0xffffffffu;
#pragma unroll
          for (int kb = 0; kb < 8; ++kb) {
            m = umin3(m, (unsigned)a[kb][0], (unsigned)a[kb][1]);
            m = umin3(m, (unsigned)a[kb][2], (unsigned)a[kb][3]);
            asm volatile("" : "+v"(m));
            const v4i av = a[kb] ^ XORK;
#pragma unroll
            for (int cb = 0; cb < 4; ++cb) acc[cb] = __builtin_amdgcn_mfma_i32_16x16x64_i8(av, B[kb][cb], acc[cb], 0, 0, 0);
          }
          if (flags_ok || __builtin_amdgcn_ballot_w64(m == 0u) == 0ull) break;
        }
        if (aggr) {
          bits = __builtin_amdgcn_ballot_w64(pv >= pref);
          flags_ok = (unsigned)(bits >> 32) == 0xffffffffu;
        }
        if (((unsigned)bits & 4u) || ((iter & 15u) == 0u && __builtin_amdgcn_s_memrealtime() - t0 > DEADLINE_TICKS)) {
          __hip_atomic_store(tmo, call, __ATOMIC_RELAXED, __HIP_MEMORY_SCOPE_AGENT);
          bad_flag = 1;
#pragma unroll
          for (int cb = 0; cb < 4; ++cb) acc[cb] = (v4i){0, 0, 0, 0};
          break;
        }
        __builtin_amdgcn_s_sleep(1);
      }
    }
    v4i* rw = &red[t & 1][wave][0];
#pragma unroll
    for (int i = 0; i < 4; ++i) rw[(4 * g + i) * 16 + c] = (v4i){acc[0][i], acc[1][i], acc[2][i], acc[3][i]};
    asm volatile("s_waitcnt lgkmcnt(0)" ::: "memory");
    __builtin_amdgcn_s_barrier();
    asm volatile("" ::: "memory");
    if (wave < 4) {
      v4i s = red[t & 1][0][wave * 64 + lane];
#pragma unroll
      for (int w2 = 1; w2 < NWAVE; ++w2) s += red[t & 1][w2][wave * 64 + lane];
      v4f o;
#pragma unroll
      for (int i = 0; i < 4; ++i) o[i] = silu_dq(s[i], scale);
      if (bad_flag) o = (v4f){__builtin_nanf(""), __builtin_nanf(""), __builtin_nanf(""), __builtin_nanf("")};
      const int row = src * M_SH + rb * 16 + wave * 4 + g;
      __builtin_amdgcn_raw_buffer_store_b128(__builtin_bit_cast(v4u, o), rsO, (row * N_SH + panel * 64 + c * 4) * 4, 0, O_AUX);
    }
  }
}

}

extern "C" void kernel_launch(int rank, int nranks, ncclComm_t comm, hipStream_t stream,
                              void* const* d_in, const int* in_sizes, int n_in,
                              void* d_out, int out_size,
                              const int* in_dtypes, int out_dtype,
                              void* const* d_out_all,
                              void* const* d_ws_all, size_t ws_bytes) {
  (void)comm; (void)in_sizes; (void)in_dtypes; (void)out_dtype; (void)d_out_all;
  if (nranks != NR || (unsigned)rank >= (unsigned)NR || n_in < 4 || ws_bytes < WS_NEED ||
      out_size != NR * M_SH * N_SH) {
    fprintf(stderr, "unexpected problem configuration\n");
    return;
  }
  Params p;
  for (int j = 0; j < 2 * (NR - 1); ++j) p.peer[j] = (char*)d_ws_all[(rank + 1 + j % (NR - 1)) % NR];
  p.ws = (char*)d_ws_all[rank];
  p.x = (const signed char*)d_in[0];
  p.w = (const signed char*)d_in[1];
  p.sx = (const float*)d_in[2];
  p.sw = (const float*)d_in[3];
  p.out = (float*)d_out;
  p.rank = rank;
  hipLaunchKernelGGL(ag_gemm_kernel, dim3(NWG), dim3(512), 0, stream, p);
  HIP_CHECK(hipGetLastError());
}
```

```cpp
#include <hip/hip_runtime.h>
#include <rccl/rccl.h>
#include <cstdio>
#include <cstddef>

#define HIP_CHECK(expr) do { hipError_t _e = (expr); if (_e != hipSuccess) { \
    fprintf(stderr, "HIP error %s at %s:%d\n", hipGetErrorString(_e), __FILE__, __LINE__); \
    return; } } while (0)

#define SEND_W 16
#define A_AUX 17
#define O_AUX 17
#define Z_AUX 17
#define AGGR_RB 31
#define Z_DELAY 300
#define Z_PACE 8

namespace {

typedef int v4i __attribute__((ext_vector_type(4)));
typedef unsigned v4u __attribute__((ext_vector_type(4)));
typedef float v4f __attribute__((ext_vector_type(4)));

constexpr int NR = 8;
constexpr int M_SH = 512, K_DIM = 4096, N_SH = 512;
constexpr size_t SHARD = (size_t)M_SH * K_DIM;
constexpr int UNITS = 512;
constexpr size_t BUF_BYTES = (size_t)NR * SHARD;
constexpr size_t STAGE_OFF = 0;
constexpr size_t FLAG_OFF = 3 * BUF_BYTES;
constexpr size_t TKT_OFF = FLAG_OFF + (size_t)NR * UNITS * 4;
constexpr size_t CTR_OFF = TKT_OFF + 128;
constexpr int NWG = 256, NWAVE = 8;
constexpr size_t TMO_OFF = CTR_OFF + (size_t)NWG * NWAVE * 4;
constexpr size_t DBG_OFF = TMO_OFF + 128;
constexpr size_t DBG_WORDS = (size_t)NWG * NWAVE * 16;
constexpr size_t WS_NEED = DBG_OFF + DBG_WORDS * 8;
constexpr unsigned long long DEADLINE_TICKS = 100000000ull;
constexpr int NGROUP = 24;
constexpr int XORK = (int)0x80808080u;

struct Params {
  char* peer[2 * (NR - 1)];
  char* ws;
  const signed char* x;
  const signed char* w;
  const float* sx;
  const float* sw;
  float* out;
  int rank;
};

#define WAITVM(n) asm volatile("s_waitcnt vmcnt(%0)" ::"n"(n) : "memory")

__device__ __forceinline__ unsigned unit_src(unsigned u) { return (u >> 4) * 65536u + (u & 15u) * 256u; }

#define X_ONE "1"
#define X_ALL "-1"
#define S_PICK                                                                       \
  "s_cmp_ge_u32 %[me], %[w1]\n\t"                                                    \
  "s_cbranch_scc1 .Lsend_skip_%=\n\t"
#define R_WS "s[48:49]"
#define R_X "s[50:51]"
#define D0 "s[56:57]"
#define D1 "s[58:59]"
#define D2 "s[60:61]"
#define D3 "s[62:63]"
#define D4 "s[64:65]"
#define D5 "s[66:67]"
#define D6 "s[68:69]"
#define S_ST(D, R, OD) "global_store_dwordx4 v67, " R ", " D " offset:" OD " sc0 sc1\n\t"
#define S_XOR(R) "v_xor_b32 " R ", %[xk], " R "\n\t"
#define S_FRAG(R0, R1, R2, R3, R, OD, OS, PRE)                                       \
  S_XOR(R0) S_XOR(R1) S_XOR(R2) S_XOR(R3)                                            \
  "v_min3_u32 v69, v69, " R0 ", " R1 "\n\t"                                          \
  "v_min3_u32 v69, v69, " R2 ", " R3 "\n\t"                                          \
  S_ST(D0, R, OD) S_ST(D1, R, OD) S_ST(D2, R, OD) S_ST(D3, R, OD)            \
  S_ST(D4, R, OD) S_ST(D5, R, OD) S_ST(D6, R, OD) PRE                          \
  "global_load_dwordx4 " R ", v66, " R_X " offset:" OS "\n\t"
#define S_W54 "s_waitcnt vmcnt(54)\n\t"
#define S_HEAD(TK, U)                                                                \
  "s_waitcnt vmcnt(53)\n\t"                                                          \
  "v_readfirstlane_b32 s43, " TK "\n\t"                                              \
  "s_mov_b64 exec, " X_ONE "\n\t"                                                    \
  "global_atomic_add " TK ", %[vtk], %[one], " R_WS " sc0\n\t"                          \
  "s_mov_b64 exec, " X_ALL "\n\t"                                                           \
  "v_lshl_add_u32 v67, " U ", 12, v68\n\t"                                           \
  "s_sub_u32 " U ", s43, s42\n\t"                                                    \
  "s_min_u32 s43, " U ", 511\n\t"                                                    \
  "s_lshl_b32 s44, s43, 12\n\t"                                                      \
  "s_lshl_b32 s43, s43, 8\n\t"                                                       \
  "s_and_b32 s44, s44, 0x1f0000\n\t"                                                 \
  "s_and_b32 s43, s43, 0xf00\n\t"                                                    \
  "s_or_b32 s43, s43, s44\n\t"
#define S_VS "v_or_b32 v66, s43, %[lsrc]\n\t"
#define S_FL(D) "global_store_dword v69, v70, " D " sc0 sc1\n\t"
#define S_EXC(L)                                                                     \
  "v_cmp_eq_u32 vcc, 0, v69\n\t"                                                     \
  "s_cbranch_vccz .Lsend_ok" L "_%=\n\t"                                             \
  "s_waitcnt vmcnt(0)\n\t"                                                           \
  "v_sub_u32 v69, v67, v68\n\t"                                                      \
  "v_lshrrev_b32 v69, 10, v69\n\t"                                                   \
  "v_add_u32 v69, s52, v69\n\t"                                                    \
  "s_mov_b64 exec, 1\n\t"                                                            \
  S_FL(D0) S_FL(D1) S_FL(D2) S_FL(D3) S_FL(D4) S_FL(D5) S_FL(D6)       \
  "s_mov_b64 exec, -1\n\t"                                                           \
  "s_waitcnt vmcnt(0)\n\t"                                                           \
  "v_mov_b32 v69, -1\n"                                                              \
  ".Lsend_ok" L "_%=:\n\t"

#define S_EXIT "s_endpgm\n"
#define S_LD(R, V, O) "global_load_dwordx4 " R ", " V ", " R_X " offset:" O "\n\t"

template <int W>
__device__ __forceinline__ unsigned sender(int v, int wave, int lane) {
  static_assert(UNITS == 512 && BUF_BYTES == (1u << 24) && STAGE_OFF == 0 && SHARD == (1u << 21), "asm constants");
  static_assert(offsetof(Params, x) == offsetof(Params, ws) + 8 && offsetof(Params, peer) == 0, "asm constants");
  const unsigned lsrc = (unsigned)((lane & 15) * 4096 + (lane >> 4) * 16);
  const unsigned vsA = lsrc + unit_src((unsigned)v), vsB = lsrc + unit_src((unsigned)(v + W));
  const unsigned vctr = (unsigned)CTR_OFF + (unsigned)(v * NWAVE * 4);
  const unsigned long long ka = (unsigned long long)__builtin_amdgcn_kernarg_segment_ptr();
  const unsigned long long kp = ka + (unsigned)(v % (NR - 1)) * 8u;
  unsigned me = (unsigned)v | (unsigned)wave << 16;
  asm volatile(
      S_PICK
      "s_load_dwordx4 s[48:51], %[ka], %[ows]\n\t"
      "s_load_dword s52, %[ka], %[ork]\n\t"
      "s_load_dwordx8 s[56:63], %[kp], 0x0\n\t"
      "s_load_dwordx4 s[64:67], %[kp], 0x20\n\t"
      "s_load_dwordx2 s[68:69], %[kp], 0x30\n\t"
      "s_mov_b32 s40, %[me]\n\t"
      "s_add_u32 s41, %[me], %[w1]\n\t"
      "v_mov_b32 v69, -1\n\t"
      "s_waitcnt lgkmcnt(0)\n\t"
      "global_load_dword v66, %[vctr], " R_WS " sc1\n\t"
      S_LD("v[32:35]", "%[vsA]", "0") S_LD("v[36:39]", "%[vsA]", "64")
      S_LD("v[40:43]", "%[vsA]", "128") S_LD("v[44:47]", "%[vsA]", "192")
      S_LD("v[48:51]", "%[vsB]", "0") S_LD("v[52:55]", "%[vsB]", "64")
      S_LD("v[56:59]", "%[vsB]", "128") S_LD("v[60:63]", "%[vsB]", "192")
      "v_lshl_or_b32 v68, s52, 21, %[l16]\n\t"
      "s_lshl_b32 s52, s52, 11\n\t"
      "s_add_u32 s52, s52, %[fo]\n\t"
      "s_waitcnt vmcnt(8)\n\t"
      "v_readfirstlane_b32 s43, v66\n\t"
      "s_add_u32 s45, s43, 1\n\t"
      "s_lshl_b32 s42, s43, 9\n\t"
      "s_sub_u32 s42, s42, %[w4]\n\t"
      "s_mul_hi_u32 s43, s45, 0xaaaaaaab\n\t"
      "s_lshr_b32 s43, s43, 1\n\t"
      "s_mul_i32 s43, s43, 3\n\t"
      "s_sub_u32 s43, s45, s43\n\t"
      "s_lshl_b32 s43, s43, 24\n\t"
      "v_add_u32 v68, s43, v68\n\t"
      "s_add_u32 s43, s42, s40\n\t"
      "s_add_u32 s44, s43, %[w3]\n\t"
      "s_add_u32 s43, s43, %[w2]\n\t"
      "v_mov_b32 v64, s43\n\t"
      "v_mov_b32 v65, s44\n\t"
      "v_mov_b32 v70, s45\n\t"
      "s_waitcnt vmcnt(0)\n\t"
      "s_mov_b64 exec, 1\n\t"
      "global_store_dword %[vctr], v70, " R_WS " sc1\n\t"
      "s_mov_b64 exec, -1\n"
      ".Lsend_loop_%=:\n\t"
      S_HEAD("v64", "s40")
      S_FRAG("v32", "v33", "v34", "v35", "v[32:35]", "0", "0", S_VS) S_W54
      S_FRAG("v36", "v37", "v38", "v39", "v[36:39]", "1024", "64", "") S_W54
      S_FRAG("v40", "v41", "v42", "v43", "v[40:43]", "2048", "128", "") S_W54
      S_FRAG("v44", "v45", "v46", "v47", "v[44:47]", "3072", "192", "")
      S_EXC("A")
      "s_cmp_gt_u32 s41, 511\n\t"
      "s_cbranch_scc1 .Lsend_end_%=\n\t"
      S_HEAD("v65", "s41")
      S_FRAG("v48", "v49", "v50", "v51", "v[48:51]", "0", "0", S_VS) S_W54
      S_FRAG("v52", "v53", "v54", "v55", "v[52:55]", "1024", "64", "") S_W54
      S_FRAG("v56", "v57", "v58", "v59", "v[56:59]", "2048", "128", "") S_W54
      S_FRAG("v60", "v61", "v62", "v63", "v[60:63]", "3072", "192", "")
      S_EXC("B")
      "s_cmp_le_u32 s40, 511\n\t"
      "s_cbranch_scc1 .Lsend_loop_%=\n"
      ".Lsend_end_%=:\n\t"
      "s_mov_b64 exec, -1\n\t"
      "s_waitcnt vmcnt(0)\n\t"
      S_EXIT
      ".Lsend_skip_%=:"
      : [me] "+s"(me)
      : [ka] "s"(ka), [kp] "s"(kp), [xk] "s"(XORK), [vctr] "v"(vctr), [vtk] "v"((unsigned)TKT_OFF),
        [one] "v"(1u), [lsrc] "v"(lsrc), [l16] "v"((unsigned)(lane * 16)), [vsA] "v"(vsA), [vsB] "v"(vsB),
        [w1] "n"(W), [w2] "n"(2 * W), [w3] "n"(3 * W), [w4] "n"(4 * W), [fo] "n"((unsigned)FLAG_OFF),
        [ows] "n"(offsetof(Params, ws)), [ork] "n"(offsetof(Params, rank))
      : "memory", "scc", "vcc", "s40", "s41", "s42", "s43", "s44", "s45", "s48", "s49", "s50", "s51", "s52",
        "s56", "s57", "s58", "s59", "s60", "s61", "s62", "s63", "s64", "s65", "s66", "s67", "s68", "s69",
        "v32", "v33", "v34", "v35", "v36", "v37", "v38", "v39", "v40", "v41", "v42", "v43", "v44", "v45", "v46", "v47",
        "v48", "v49", "v50", "v51", "v52", "v53", "v54", "v55", "v56", "v57", "v58", "v59", "v60", "v61", "v62", "v63",
        "v64", "v65", "v66", "v67", "v68", "v69", "v70");
  return me;
}

__device__ __forceinline__ unsigned perm(unsigned hi, unsigned lo, unsigned sel) {
  return __builtin_amdgcn_perm(hi, lo, sel);
}

__device__ __forceinline__ float silu_dq(int acc, float scale) {
  const float y = (float)acc * scale;
  const float yc = fminf(fmaxf(y, -60.0f), 60.0f);
  return y * __builtin_amdgcn_rcpf(1.0f + __expf(-yc));
}

__device__ __forceinline__ unsigned umin3(unsigned a, unsigned b, unsigned c) { return min(min(a, b), c); }

__global__ void __launch_bounds__(512, 2) ag_gemm_kernel(Params p) {
  __shared__ v4i red[2][NWAVE][256];
  __shared__ int bad_flag;
  const int b = blockIdx.x, tid = threadIdx.x, lane = tid & 63;
  const int wave = __builtin_amdgcn_readfirstlane(tid >> 6);
  const unsigned me = sender<SEND_W>(b, wave, lane);
  (void)me;
  const int xcd = b & 7, idx = b >> 3;
  unsigned* myctr = (unsigned*)(p.ws + CTR_OFF) + b * NWAVE + wave;

  if (idx < 8) {
    if (b >= SEND_W) {
      const unsigned long long tz = __builtin_amdgcn_s_memrealtime();
      const unsigned cprev = __hip_atomic_load(myctr, __ATOMIC_RELAXED, __HIP_MEMORY_SCOPE_AGENT);
      const unsigned call = __builtin_amdgcn_readfirstlane(cprev) + 1u;
      const __amdgpu_buffer_rsrc_t rsZ =
          __builtin_amdgcn_make_buffer_rsrc(p.ws + STAGE_OFF + (size_t)((call + 2u) % 3u) * BUF_BYTES, 0, (int)BUF_BYTES, 0x00020000);
      if (lane == 0) __hip_atomic_store(myctr, call, __ATOMIC_RELAXED, __HIP_MEMORY_SCOPE_AGENT);
      constexpr int NZ = (64 - SEND_W) * NWAVE;
      while (__builtin_amdgcn_s_memrealtime() - tz < Z_DELAY) __builtin_amdgcn_s_sleep(8);
      for (int ch = (b - SEND_W) * NWAVE + wave; ch < 7 * 2048; ch += NZ) {
        const int off = ((p.rank + 1 + (ch >> 11)) & 7) * (int)SHARD + (ch & 2047) * 1024 + lane * 16;
        __builtin_amdgcn_raw_buffer_store_b128((v4u){0u, 0u, 0u, 0u}, rsZ, off, 0, Z_AUX);
        __builtin_amdgcn_s_sleep(Z_PACE);
      }
    }
    return;
  }

  const int cidx = idx - 8;
  const int grp = (cidx >> 3) * 8 + xcd;
  const int panel = cidx & 7;
  const int c = lane & 15, g = lane >> 4;
  const unsigned cprev = __hip_atomic_load(myctr, __ATOMIC_RELAXED, __HIP_MEMORY_SCOPE_AGENT);
  if (tid == 0) bad_flag = 0;

  v4i B[8][4];
  {
    const signed char* wp = p.w + (size_t)(wave * 512 + g * 16) * N_SH + panel * 64 + c * 4;
#pragma unroll
    for (int kb = 0; kb < 8; ++kb) {
      unsigned r[16];
#pragma unroll
      for (int j = 0; j < 16; ++j) r[j] = *(const unsigned*)(wp + (size_t)(kb * 64 + j) * N_SH);
#pragma unroll
      for (int q = 0; q < 4; ++q) {
        const unsigned t0 = perm(r[4 * q + 1], r[4 * q + 0], 0x05010400u);
        const unsigned t1 = perm(r[4 * q + 1], r[4 * q + 0], 0x07030602u);
        const unsigned t2 = perm(r[4 * q + 3], r[4 * q + 2], 0x05010400u);
        const unsigned t3 = perm(r[4 * q + 3], r[4 * q + 2], 0x07030602u);
        B[kb][0][q] = (int)perm(t2, t0, 0x05040100u);
        B[kb][1][q] = (int)perm(t2, t0, 0x07060302u);
        B[kb][2][q] = (int)perm(t3, t1, 0x05040100u);
        B[kb][3][q] = (int)perm(t3, t1, 0x07060302u);
      }
    }
  }
  const float scale = p.sx[0] * p.sw[0];
  const unsigned call = __builtin_amdgcn_readfirstlane(cprev) + 1u;
  if (lane == 0) __hip_atomic_store(myctr, call, __ATOMIC_RELAXED, __HIP_MEMORY_SCOPE_AGENT);
  unsigned* tmo = (unsigned*)(p.ws + TMO_OFF);
  const unsigned bufoff = (unsigned)(call % 3u) * (unsigned)BUF_BYTES;
  const __amdgpu_buffer_rsrc_t rsW = __builtin_amdgcn_make_buffer_rsrc(p.ws, 0, (int)DBG_OFF, 0x00020000);
  const __amdgpu_buffer_rsrc_t rsX = __builtin_amdgcn_make_buffer_rsrc((void*)p.x, 0, (int)SHARD, 0x00020000);
  const __amdgpu_buffer_rsrc_t rsO = __builtin_amdgcn_make_buffer_rsrc((void*)p.out, 0, NR * M_SH * N_SH * 4, 0x00020000);
  __syncthreads();

  const int nlocal = (grp >= 8 && grp < 16) ? 2 : 1;
  const int nremote = (grp < 8) ? 10 : 9;
  const int ntask = nlocal + nremote;
  for (int t = 0; t < ntask; ++t) {
    int src, rb;
    v4i acc[4];
    if (t < nlocal) {
      src = p.rank;
      rb = (t == 0) ? grp : 16 + grp;
      const int off = (rb * 16 + c) * K_DIM + wave * 512 + g * 16;
      v4i a[8];
#pragma unroll
      for (int kb = 0; kb < 8; ++kb)
        a[kb] = __builtin_bit_cast(v4i, __builtin_amdgcn_raw_buffer_load_b128(rsX, off + kb * 64, 0, 0));
      __builtin_amdgcn_sched_barrier(0);
#pragma unroll
      for (int cb = 0; cb < 4; ++cb) acc[cb] = (v4i){0, 0, 0, 0};
#pragma unroll
      for (int kb = 0; kb < 8; ++kb)
#pragma unroll
        for (int cb = 0; cb < 4; ++cb) acc[cb] = __builtin_amdgcn_mfma_i32_16x16x64_i8(a[kb], B[kb][cb], acc[cb], 0, 0, 0);
    } else {
      const int n = grp + NGROUP * (t - nlocal);
      rb = n / 7;
      src = (p.rank + 1 + (n - rb * 7)) & 7;
      const unsigned base = bufoff + (unsigned)src * (unsigned)SHARD + (unsigned)(rb * 65536 + wave * 8192);
      const int off = (int)base + lane * 16;
      const int poff = (lane >= 4) ? (int)TMO_OFF
                       : (lane >= 2) ? (int)(FLAG_OFF + (unsigned)(src * UNITS + rb * 16 + wave * 2 + (lane & 1)) * 4u)
                                     : (int)(base + 4092u + (lane & 1) * 4096u);
      const unsigned pref = (lane >= 2) ? call : 1u;
      const bool aggr = rb >= AGGR_RB;
      unsigned fl = 0;
      const unsigned long long t0 = __builtin_amdgcn_s_memrealtime();
      for (unsigned iter = 1;; ++iter) {
        const unsigned pv = __builtin_amdgcn_raw_buffer_load_b32(rsW, poff, 0, 17);
        bool go = true;
        unsigned bits = 0;
        if (!aggr) {
          bits = (unsigned)__builtin_amdgcn_ballot_w64(pv >= pref);
          fl = bits >> 2;
          go = ((bits | fl) & 3u) == 3u;
        }
        if (go) {
          v4i a[8];
#pragma unroll
          for (int kb = 0; kb < 8; ++kb)
            a[kb] = __builtin_bit_cast(v4i, __builtin_amdgcn_raw_buffer_load_b128(rsW, off + kb * 1024, 0, A_AUX));
          __builtin_amdgcn_sched_barrier(0);
#pragma unroll
          for (int cb = 0; cb < 4; ++cb) acc[cb] = (v4i){0, 0, 0, 0};
          unsigned m = 0xffffffffu, m0 = 0;
#pragma unroll
          for (int kb = 0; kb < 8; ++kb) {
            if (kb == 4) { m0 = m | (fl & 1u); m = 0xffffffffu; }
            m = umin3(m, (unsigned)a[kb][0], (unsigned)a[kb][1]);
            m = umin3(m, (unsigned)a[kb][2], (unsigned)a[kb][3]);
            asm volatile("" : "+v"(m));
            const v4i av = a[kb] ^ XORK;
#pragma unroll
            for (int cb = 0; cb < 4; ++cb) acc[cb] = __builtin_amdgcn_mfma_i32_16x16x64_i8(av, B[kb][cb], acc[cb], 0, 0, 0);
          }
          if (__builtin_amdgcn_ballot_w64(min(m0, m | (fl & 2u)) == 0u) == 0ull) break;
        }
        if (aggr) {
          bits = (unsigned)__builtin_amdgcn_ballot_w64(pv >= pref);
          fl = bits >> 2;
        }
        if ((bits & 16u) || ((iter & 15u) == 0u && __builtin_amdgcn_s_memrealtime() - t0 > DEADLINE_TICKS)) {
          __hip_atomic_store(tmo, call, __ATOMIC_RELAXED, __HIP_MEMORY_SCOPE_AGENT);
          bad_flag = 1;
#pragma unroll
          for (int cb = 0; cb < 4; ++cb) acc[cb] = (v4i){0, 0, 0, 0};
          break;
        }
        __builtin_amdgcn_s_sleep(1);
      }
    }
    v4i* rw = &red[t & 1][wave][0];
#pragma unroll
    for (int i = 0; i < 4; ++i) rw[(4 * g + i) * 16 + c] = (v4i){acc[0][i], acc[1][i], acc[2][i], acc[3][i]};
    asm volatile("s_waitcnt lgkmcnt(0)" ::: "memory");
    __builtin_amdgcn_s_barrier();
    asm volatile("" ::: "memory");
    if (wave < 4) {
      v4i s = red[t & 1][0][wave * 64 + lane];
#pragma unroll
      for (int w2 = 1; w2 < NWAVE; ++w2) s += red[t & 1][w2][wave * 64 + lane];
      v4f o;
#pragma unroll
      for (int i = 0; i < 4; ++i) o[i] = silu_dq(s[i], scale);
      if (bad_flag) o = (v4f){__builtin_nanf(""), __builtin_nanf(""), __builtin_nanf(""), __builtin_nanf("")};
      const int row = src * M_SH + rb * 16 + wave * 4 + g;
      __builtin_amdgcn_raw_buffer_store_b128(__builtin_bit_cast(v4u, o), rsO, (row * N_SH + panel * 64 + c * 4) * 4, 0, O_AUX);
    }
  }
}

}

extern "C" void kernel_launch(int rank, int nranks, ncclComm_t comm, hipStream_t stream,
                              void* const* d_in, const int* in_sizes, int n_in,
                              void* d_out, int out_size,
                              const int* in_dtypes, int out_dtype,
                              void* const* d_out_all,
                              void* const* d_ws_all, size_t ws_bytes) {
  (void)comm; (void)in_sizes; (void)in_dtypes; (void)out_dtype; (void)d_out_all;
  if (nranks != NR || (unsigned)rank >= (unsigned)NR || n_in < 4 || ws_bytes < WS_NEED ||
      out_size != NR * M_SH * N_SH) {
    fprintf(stderr, "unexpected problem configuration\n");
    return;
  }
  Params p;
  for (int j = 0; j < 2 * (NR - 1); ++j) p.peer[j] = (char*)d_ws_all[(rank + 1 + j % (NR - 1)) % NR];
  p.ws = (char*)d_ws_all[rank];
  p.x = (const signed char*)d_in[0];
  p.w = (const signed char*)d_in[1];
  p.sx = (const float*)d_in[2];
  p.sw = (const float*)d_in[3];
  p.out = (float*)d_out;
  p.rank = rank;
  hipLaunchKernelGGL(ag_gemm_kernel, dim3(NWG), dim3(512), 0, stream, p);
  HIP_CHECK(hipGetLastError());
}
```

```cpp
#include <hip/hip_runtime.h>
#include <rccl/rccl.h>
#include <cstdio>
#include <cstddef>

#define HIP_CHECK(expr) do { hipError_t _e = (expr); if (_e != hipSuccess) { \
    fprintf(stderr, "HIP error %s at %s:%d\n", hipGetErrorString(_e), __FILE__, __LINE__); \
    return; } } while (0)

#define SEND_W 16
#define A_AUX 17
#define O_AUX 17
#define Z_AUX 17
#define AGGR_RB 31
#define GATE 1
#define Z_DELAY 300
#define Z_PACE 8

namespace {

typedef int v4i __attribute__((ext_vector_type(4)));
typedef unsigned v4u __attribute__((ext_vector_type(4)));
typedef float v4f __attribute__((ext_vector_type(4)));

constexpr int NR = 8;
constexpr int M_SH = 512, K_DIM = 4096, N_SH = 512;
constexpr size_t SHARD = (size_t)M_SH * K_DIM;
constexpr int UNITS = 512;
constexpr size_t BUF_BYTES = (size_t)NR * SHARD;
constexpr size_t STAGE_OFF = 0;
constexpr size_t FLAG_OFF = 3 * BUF_BYTES;
constexpr size_t TKT_OFF = FLAG_OFF + (size_t)NR * UNITS * 4;
constexpr size_t CTR_OFF = TKT_OFF + 128;
constexpr int NWG = 256, NWAVE = 8;
constexpr size_t TMO_OFF = CTR_OFF + (size_t)NWG * NWAVE * 4;
constexpr size_t DBG_OFF = TMO_OFF + 128;
constexpr size_t DBG_WORDS = (size_t)NWG * NWAVE * 16;
constexpr size_t WS_NEED = DBG_OFF + DBG_WORDS * 8;
constexpr unsigned long long DEADLINE_TICKS = 100000000ull;
constexpr int NGROUP = 24;
static_assert(GATE >= 0 && GATE <= AGGR_RB, "gate block");
constexpr int XORK = (int)0x80808080u;

struct Params {
  char* peer[2 * (NR - 1)];
  char* ws;
  const signed char* x;
  const signed char* w;
  const float* sx;
  const float* sw;
  float* out;
  int rank;
};

#define WAITVM(n) asm volatile("s_waitcnt vmcnt(%0)" ::"n"(n) : "memory")

#define FIRST_W "7"
#define X_ONE "1"
#define X_ALL "-1"
#define R_WS "s[48:49]"
#define R_X "s[50:51]"
#define D0 "s[56:57]"
#define D1 "s[58:59]"
#define D2 "s[60:61]"
#define D3 "s[62:63]"
#define D4 "s[64:65]"
#define D5 "s[66:67]"
#define D6 "s[68:69]"
#define S_ST(D, R, OD) "global_store_dwordx4 v67, " R ", " D " offset:" OD " sc0 sc1\n\t"
#define S_XOR(R) "v_xor_b32 " R ", s53, " R "\n\t"
#define S_FRAG(R0, R1, R2, R3, R, OD, OS, PRE)                                       \
  S_XOR(R0) S_XOR(R1) S_XOR(R2) S_XOR(R3)                                            \
  "v_min3_u32 v69, v69, " R0 ", " R1 "\n\t"                                          \
  "v_min3_u32 v69, v69, " R2 ", " R3 "\n\t"                                          \
  S_ST(D0, R, OD) S_ST(D1, R, OD) S_ST(D2, R, OD) S_ST(D3, R, OD)            \
  S_ST(D4, R, OD) S_ST(D5, R, OD) S_ST(D6, R, OD) PRE                          \
  "global_load_dwordx4 " R ", v66, " R_X " offset:" OS "\n\t"
#define S_W54 "s_waitcnt vmcnt(54)\n\t"
#define S_W1(N) S_W54 "s_cbranch_vccz .Lsend_w" N "_%=\n\ts_waitcnt vmcnt(" N ")\n.Lsend_w" N "_%=:\n\t"
#define S_HEAD(TK, U)                                                                \
  "s_waitcnt vmcnt(53)\n\t"                                                          \
  "v_readfirstlane_b32 s43, " TK "\n\t"                                              \
  "s_mov_b64 exec, " X_ONE "\n\t"                                                    \
  "global_atomic_add " TK ", v72, v73, " R_WS " sc0\n\t"                          \
  "s_mov_b64 exec, " X_ALL "\n\t"                                                           \
  "v_lshl_add_u32 v67, " U ", 12, v68\n\t"                                           \
  "s_sub_u32 " U ", s43, s42\n\t"                                                    \
  "s_min_u32 s43, " U ", 511\n\t"                                                    \
  "s_lshl_b32 s44, s43, 12\n\t"                                                      \
  "s_lshl_b32 s43, s43, 8\n\t"                                                       \
  "s_and_b32 s44, s44, 0x1f0000\n\t"                                                 \
  "s_and_b32 s43, s43, 0xf00\n\t"                                                    \
  "s_or_b32 s43, s43, s44\n\t"
#define S_VS "v_or_b32 v66, s43, v71\n\t"
#define S_FL(D) "global_store_dword v69, v70, " D " sc0 sc1\n\t"
#define S_EXC(L)                                                                     \
  "v_cmp_eq_u32 vcc, 0, v69\n\t"                                                     \
  "s_cbranch_vccz .Lsend_ok" L "_%=\n\t"                                             \
  "s_waitcnt vmcnt(0)\n\t"                                                           \
  "v_sub_u32 v69, v67, v68\n\t"                                                      \
  "v_lshrrev_b32 v69, 10, v69\n\t"                                                   \
  "v_add_u32 v69, s52, v69\n\t"                                                    \
  "s_mov_b64 exec, 1\n\t"                                                            \
  S_FL(D0) S_FL(D1) S_FL(D2) S_FL(D3) S_FL(D4) S_FL(D5) S_FL(D6)       \
  "s_mov_b64 exec, -1\n\t"                                                           \
  "s_waitcnt vmcnt(0)\n\t"                                                           \
  "v_mov_b32 v69, -1\n"                                                              \
  ".Lsend_ok" L "_%=:\n\t"

#define S_EXIT "s_endpgm\n"
#define S_LD(R, V, O) "global_load_dwordx4 " R ", " V ", " R_X " offset:" O "\n\t"

template <int W>
__device__ __forceinline__ unsigned sender(unsigned me, unsigned tid) {
  static_assert(UNITS == 512 && BUF_BYTES == (1u << 24) && STAGE_OFF == 0 && SHARD == (1u << 21), "asm constants");
  static_assert(offsetof(Params, x) == offsetof(Params, ws) + 8 && offsetof(Params, peer) == 0, "asm constants");
  static_assert(W == 16 && NR == 8 && NWAVE == 8 && K_DIM == 4096, "asm constants");
  const unsigned long long ka = (unsigned long long)__builtin_amdgcn_kernarg_segment_ptr();
  asm volatile(
      "s_cmp_ge_u32 %[me], %[w1]\n\t"
      "s_cbranch_scc1 .Lsend_skip_%=\n\t"
      "s_load_dwordx4 s[48:51], %[ka], %[ows]\n\t"
      "s_load_dword s52, %[ka], %[ork]\n\t"
      "s_mul_i32 s44, %[me], 37\n\t"
      "s_lshr_b32 s44, s44, 8\n\t"
      "s_mul_i32 s44, s44, 7\n\t"
      "s_sub_u32 s44, %[me], s44\n\t"
      "s_lshl_b32 s44, s44, 3\n\t"
      "s_load_dwordx8 s[56:63], %[ka], s44\n\t"
      "s_load_dwordx4 s[64:67], %[ka], s44 offset:0x20\n\t"
      "s_load_dwordx2 s[68:69], %[ka], s44 offset:0x30\n\t"
      "v_readfirstlane_b32 s43, %[tid]\n\t"
      "v_and_b32 v71, 15, %[tid]\n\t"
      "v_and_b32 v64, 48, %[tid]\n\t"
      "s_cmp_ge_u32 s43, 64\n\t"
      "s_cbranch_scc1 .Lsend_end_%=\n\t"
      "v_lshl_or_b32 v71, v71, 12, v64\n\t"
      "s_lshl_b32 s43, %[me], 8\n\t"
      "v_or_b32 v64, s43, v71\n\t"
      "v_add_u32 v65, 0x10000, v64\n\t"
      "s_lshl_b32 s43, %[me], 5\n\t"
      "s_add_u32 s43, s43, %[co]\n\t"
      "v_mov_b32 v74, s43\n\t"
      "v_mov_b32 v72, %[to]\n\t"
      "v_mov_b32 v73, 1\n\t"
      "s_mov_b32 s53, 0x80808080\n\t"
      "v_lshlrev_b32 v68, 4, %[tid]\n\t"
      "s_mov_b32 s40, %[me]\n\t"
      "s_add_u32 s41, %[me], %[w1]\n\t"
      "v_mov_b32 v69, -1\n\t"
      "s_mov_b64 vcc, -1\n\t"
      "s_waitcnt lgkmcnt(0)\n\t"
      "global_load_dword v66, v74, " R_WS " sc1\n\t"
      S_LD("v[32:35]", "v64", "0") S_LD("v[36:39]", "v64", "64")
      S_LD("v[40:43]", "v64", "128") S_LD("v[44:47]", "v64", "192")
      S_LD("v[48:51]", "v65", "0") S_LD("v[52:55]", "v65", "64")
      S_LD("v[56:59]", "v65", "128") S_LD("v[60:63]", "v65", "192")
      "v_lshl_or_b32 v68, s52, 21, v68\n\t"
      "s_lshl_b32 s52, s52, 11\n\t"
      "s_add_u32 s52, s52, %[fo]\n\t"
      "s_waitcnt vmcnt(8)\n\t"
      "v_readfirstlane_b32 s43, v66\n\t"
      "s_add_u32 s45, s43, 1\n\t"
      "s_lshl_b32 s42, s43, 9\n\t"
      "s_sub_u32 s42, s42, %[w4]\n\t"
      "s_mul_hi_u32 s43, s45, 0xaaaaaaab\n\t"
      "s_lshr_b32 s43, s43, 1\n\t"
      "s_mul_i32 s43, s43, 3\n\t"
      "s_sub_u32 s43, s45, s43\n\t"
      "s_lshl_b32 s43, s43, 24\n\t"
      "v_add_u32 v68, s43, v68\n\t"
      "s_add_u32 s43, s42, s40\n\t"
      "s_add_u32 s44, s43, %[w3]\n\t"
      "s_add_u32 s43, s43, %[w2]\n\t"
      "v_mov_b32 v64, s43\n\t"
      "v_mov_b32 v65, s44\n\t"
      "v_mov_b32 v70, s45\n\t"
      "s_waitcnt vmcnt(" FIRST_W ")\n\t"
      "s_mov_b64 exec, 1\n\t"
      "global_store_dword v74, v70, " R_WS " sc1\n\t"
      "s_mov_b64 exec, -1\n"
      ".Lsend_loop_%=:\n\t"
      S_HEAD("v64", "s40")
      S_FRAG("v32", "v33", "v34", "v35", "v[32:35]", "0", "0", S_VS) S_W1("16")
      S_FRAG("v36", "v37", "v38", "v39", "v[36:39]", "1024", "64", "") S_W1("23")
      S_FRAG("v40", "v41", "v42", "v43", "v[40:43]", "2048", "128", "") S_W1("30")
      S_FRAG("v44", "v45", "v46", "v47", "v[44:47]", "3072", "192", "")
      S_EXC("A")
      "s_cmp_gt_u32 s41, 511\n\t"
      "s_cbranch_scc1 .Lsend_end_%=\n\t"
      "s_cmp_ge_u32 s41, %[w2]\n\t"
      "s_cbranch_scc1 .Lsend_b_%=\n\t"
      "s_waitcnt vmcnt(34)\n"
      ".Lsend_b_%=:\n\t"
      S_HEAD("v65", "s41")
      S_FRAG("v48", "v49", "v50", "v51", "v[48:51]", "0", "0", S_VS) S_W54
      S_FRAG("v52", "v53", "v54", "v55", "v[52:55]", "1024", "64", "") S_W54
      S_FRAG("v56", "v57", "v58", "v59", "v[56:59]", "2048", "128", "") S_W54
      S_FRAG("v60", "v61", "v62", "v63", "v[60:63]", "3072", "192", "")
      S_EXC("B")
      "s_cmp_le_u32 s40, 511\n\t"
      "s_cbranch_scc1 .Lsend_loop_%=\n"
      ".Lsend_end_%=:\n\t"
      "s_mov_b64 exec, -1\n\t"
      "s_waitcnt vmcnt(0) lgkmcnt(0)\n\t"
      S_EXIT
      ".Lsend_skip_%=:"
      : [me] "+s"(me)
      : [ka] "s"(ka), [tid] "v"(tid), [co] "n"((unsigned)CTR_OFF), [to] "n"((unsigned)TKT_OFF),
        [w1] "n"(W), [w2] "n"(2 * W), [w3] "n"(3 * W), [w4] "n"(4 * W), [fo] "n"((unsigned)FLAG_OFF),
        [ows] "n"(offsetof(Params, ws)), [ork] "n"(offsetof(Params, rank))
      : "memory", "scc", "vcc", "s40", "s41", "s42", "s43", "s44", "s45", "s48", "s49", "s50", "s51", "s52", "s53",
        "s56", "s57", "s58", "s59", "s60", "s61", "s62", "s63", "s64", "s65", "s66", "s67", "s68", "s69",
        "v32", "v33", "v34", "v35", "v36", "v37", "v38", "v39", "v40", "v41", "v42", "v43", "v44", "v45", "v46", "v47",
        "v48", "v49", "v50", "v51", "v52", "v53", "v54", "v55", "v56", "v57", "v58", "v59", "v60", "v61", "v62", "v63",
        "v64", "v65", "v66", "v67", "v68", "v69", "v70", "v71", "v72", "v73", "v74");
  __builtin_amdgcn_sched_barrier(0);
  return me;
}

__device__ __forceinline__ unsigned perm(unsigned hi, unsigned lo, unsigned sel) {
  return __builtin_amdgcn_perm(hi, lo, sel);
}

__device__ __forceinline__ float silu_dq(int acc, float scale) {
  const float y = (float)acc * scale;
  const float yc = fminf(fmaxf(y, -60.0f), 60.0f);
  return y * __builtin_amdgcn_rcpf(1.0f + __expf(-yc));
}

__device__ __forceinline__ unsigned umin3(unsigned a, unsigned b, unsigned c) { return min(min(a, b), c); }

__global__ void __launch_bounds__(512, 2) ag_gemm_kernel(Params p) {
  __shared__ v4i red[2][NWAVE][256];
  __shared__ int bad_flag;
  const int b = blockIdx.x, tid = threadIdx.x, lane = tid & 63;
  const int wave = __builtin_amdgcn_readfirstlane(tid >> 6);
  const unsigned me = sender<SEND_W>((unsigned)b, (unsigned)tid);
  (void)me;
  const int xcd = b & 7, idx = b >> 3;
  unsigned* myctr = (unsigned*)(p.ws + CTR_OFF) + b * NWAVE + wave;

  if (idx < 8) {
    if (b >= SEND_W) {
      const unsigned long long tz = __builtin_amdgcn_s_memrealtime();
      const unsigned cprev = __hip_atomic_load(myctr, __ATOMIC_RELAXED, __HIP_MEMORY_SCOPE_AGENT);
      const unsigned call = __builtin_amdgcn_readfirstlane(cprev) + 1u;
      const __amdgpu_buffer_rsrc_t rsZ =
          __builtin_amdgcn_make_buffer_rsrc(p.ws + STAGE_OFF + (size_t)((call + 2u) % 3u) * BUF_BYTES, 0, (int)BUF_BYTES, 0x00020000);
      if (lane == 0) __hip_atomic_store(myctr, call, __ATOMIC_RELAXED, __HIP_MEMORY_SCOPE_AGENT);
      constexpr int NZ = (64 - SEND_W) * NWAVE;
      while (__builtin_amdgcn_s_memrealtime() - tz < Z_DELAY) __builtin_amdgcn_s_sleep(8);
      for (int ch = (b - SEND_W) * NWAVE + wave; ch < 7 * 2048; ch += NZ) {
        const int off = ((p.rank + 1 + (ch >> 11)) & 7) * (int)SHARD + (ch & 2047) * 1024 + lane * 16;
        __builtin_amdgcn_raw_buffer_store_b128((v4u){0u, 0u, 0u, 0u}, rsZ, off, 0, Z_AUX);
        __builtin_amdgcn_s_sleep(Z_PACE);
      }
    }
    return;
  }

  const int cidx = idx - 8;
  const int grp = (cidx >> 3) * 8 + xcd;
  const int panel = cidx & 7;
  const int c = lane & 15, g = lane >> 4;
  const unsigned cprev = __hip_atomic_load(myctr, __ATOMIC_RELAXED, __HIP_MEMORY_SCOPE_AGENT);
  if (tid == 0) bad_flag = 0;

  v4i B[8][4];
  {
    const signed char* wp = p.w + (size_t)(wave * 512 + g * 16) * N_SH + panel * 64 + c * 4;
#pragma unroll
    for (int kb = 0; kb < 8; ++kb) {
      unsigned r[16];
#pragma unroll
      for (int j = 0; j < 16; ++j) r[j] = *(const unsigned*)(wp + (size_t)(kb * 64 + j) * N_SH);
#pragma unroll
      for (int q = 0; q < 4; ++q) {
        const unsigned t0 = perm(r[4 * q + 1], r[4 * q + 0], 0x05010400u);
        const unsigned t1 = perm(r[4 * q + 1], r[4 * q + 0], 0x07030602u);
        const unsigned t2 = perm(r[4 * q + 3], r[4 * q + 2], 0x05010400u);
        const unsigned t3 = perm(r[4 * q + 3], r[4 * q + 2], 0x07030602u);
        B[kb][0][q] = (int)perm(t2, t0, 0x05040100u);
        B[kb][1][q] = (int)perm(t2, t0, 0x07060302u);
        B[kb][2][q] = (int)perm(t3, t1, 0x05040100u);
        B[kb][3][q] = (int)perm(t3, t1, 0x07060302u);
      }
    }
  }
  const float scale = p.sx[0] * p.sw[0];
  const unsigned call = __builtin_amdgcn_readfirstlane(cprev) + 1u;
  if (lane == 0) __hip_atomic_store(myctr, call, __ATOMIC_RELAXED, __HIP_MEMORY_SCOPE_AGENT);
  unsigned* tmo = (unsigned*)(p.ws + TMO_OFF);
  const unsigned bufoff = (unsigned)(call % 3u) * (unsigned)BUF_BYTES;
  const __amdgpu_buffer_rsrc_t rsW = __builtin_amdgcn_make_buffer_rsrc(p.ws, 0, (int)DBG_OFF, 0x00020000);
  const __amdgpu_buffer_rsrc_t rsX = __builtin_amdgcn_make_buffer_rsrc((void*)p.x, 0, (int)SHARD, 0x00020000);
  const __amdgpu_buffer_rsrc_t rsO = __builtin_amdgcn_make_buffer_rsrc((void*)p.out, 0, NR * M_SH * N_SH * 4, 0x00020000);
  __syncthreads();

  const int nlocal = (grp >= 8 && grp < 16) ? 2 : 1;
  const int nremote = (grp < 8) ? 10 : 9;
  const int ntask = nlocal + nremote;
  for (int t = 0; t < ntask; ++t) {
    int src, rb;
    v4i acc[4];
    if (t < nlocal) {
      src = p.rank;
      rb = (t == 0) ? grp : 16 + grp;
      const int off = (rb * 16 + c) * K_DIM + wave * 512 + g * 16;
      v4i a[8];
#pragma unroll
      for (int kb = 0; kb < 8; ++kb)
        a[kb] = __builtin_bit_cast(v4i, __builtin_amdgcn_raw_buffer_load_b128(rsX, off + kb * 64, 0, 0));
      __builtin_amdgcn_sched_barrier(0);
#pragma unroll
      for (int cb = 0; cb < 4; ++cb) acc[cb] = (v4i){0, 0, 0, 0};
#pragma unroll
      for (int kb = 0; kb < 8; ++kb)
#pragma unroll
        for (int cb = 0; cb < 4; ++cb) acc[cb] = __builtin_amdgcn_mfma_i32_16x16x64_i8(a[kb], B[kb][cb], acc[cb], 0, 0, 0);
    } else {
      const int n = grp + NGROUP * (t - nlocal);
      rb = n / 7;
      src = (p.rank + 1 + (n - rb * 7)) & 7;
      const unsigned base = bufoff + (unsigned)src * (unsigned)SHARD + (unsigned)(rb * 65536 + wave * 8192);
      const int off = (int)base + lane * 16;
      const int poff1 = (lane >= 4) ? (int)TMO_OFF
                        : (lane >= 2) ? (int)(FLAG_OFF + (unsigned)(src * UNITS + rb * 16 + wave * 2 + (lane & 1)) * 4u)
                                      : (int)(base + 4092u + (lane & 1) * 4096u);
      const unsigned pref = (lane >= 2) ? call : 1u;
      int mode = (rb >= AGGR_RB) ? (GATE ? 1 : 2) : 0;
      int poff = poff1;
      if (mode == 1) poff -= (lane >= 4) ? 0 : (lane >= 2) ? GATE * 64 : GATE * 65536;
      unsigned fl = 0;
      const unsigned long long t0 = __builtin_amdgcn_s_memrealtime();
      for (unsigned iter = 1;; ++iter) {
        const unsigned pv = __builtin_amdgcn_raw_buffer_load_b32(rsW, poff, 0, 17);
        bool go = true;
        unsigned bits = 0;
        const bool aggr = mode == 2;
        if (!aggr) {
          bits = (unsigned)__builtin_amdgcn_ballot_w64(pv >= pref);
          fl = bits >> 2;
          go = ((bits | fl) & 3u) == 3u;
          if (mode == 1) {
            fl = 0;
            if (go) { mode = 2; poff = poff1; }
          }
        }
        if (go) {
          v4i a[8];
#pragma unroll
          for (int kb = 0; kb < 8; ++kb)
            a[kb] = __builtin_bit_cast(v4i, __builtin_amdgcn_raw_buffer_load_b128(rsW, off + kb * 1024, 0, A_AUX));
          __builtin_amdgcn_sched_barrier(0);
#pragma unroll
          for (int cb = 0; cb < 4; ++cb) acc[cb] = (v4i){0, 0, 0, 0};
          unsigned m = 0xffffffffu, m0 = 0;
#pragma unroll
          for (int kb = 0; kb < 8; ++kb) {
            if (kb == 4) { m0 = m | (fl & 1u); m = 0xffffffffu; }
            m = umin3(m, (unsigned)a[kb][0], (unsigned)a[kb][1]);
            m = umin3(m, (unsigned)a[kb][2], (unsigned)a[kb][3]);
            asm volatile("" : "+v"(m));
            const v4i av = a[kb] ^ XORK;
#pragma unroll
            for (int cb = 0; cb < 4; ++cb) acc[cb] = __builtin_amdgcn_mfma_i32_16x16x64_i8(av, B[kb][cb], acc[cb], 0, 0, 0);
          }
          if (__builtin_amdgcn_ballot_w64(min(m0, m | (fl & 2u)) == 0u) == 0ull) break;
        }
        if (aggr) {
          bits = (unsigned)__builtin_amdgcn_ballot_w64(pv >= pref);
          fl = bits >> 2;
        }
        if ((bits & 16u) || ((iter & 15u) == 0u && __builtin_amdgcn_s_memrealtime() - t0 > DEADLINE_TICKS)) {
          __hip_atomic_store(tmo, call, __ATOMIC_RELAXED, __HIP_MEMORY_SCOPE_AGENT);
          bad_flag = 1;
#pragma unroll
          for (int cb = 0; cb < 4; ++cb) acc[cb] = (v4i){0, 0, 0, 0};
          break;
        }
        __builtin_amdgcn_s_sleep(1);
      }
    }
    v4i* rw = &red[t & 1][wave][0];
#pragma unroll
    for (int i = 0; i < 4; ++i) rw[(4 * g + i) * 16 + c] = (v4i){acc[0][i], acc[1][i], acc[2][i], acc[3][i]};
    asm volatile("s_waitcnt lgkmcnt(0)" ::: "memory");
    __builtin_amdgcn_s_barrier();
    asm volatile("" ::: "memory");
    if (wave < 4) {
      v4i s = red[t & 1][0][wave * 64 + lane];
#pragma unroll
      for (int w2 = 1; w2 < NWAVE; ++w2) s += red[t & 1][w2][wave * 64 + lane];
      v4f o;
#pragma unroll
      for (int i = 0; i < 4; ++i) o[i] = silu_dq(s[i], scale);
      if (bad_flag) o = (v4f){__builtin_nanf(""), __builtin_nanf(""), __builtin_nanf(""), __builtin_nanf("")};
      const int row = src * M_SH + rb * 16 + wave * 4 + g;
      __builtin_amdgcn_raw_buffer_store_b128(__builtin_bit_cast(v4u, o), rsO, (row * N_SH + panel * 64 + c * 4) * 4, 0, O_AUX);
    }
  }
}

}

extern "C" void kernel_launch(int rank, int nranks, ncclComm_t comm, hipStream_t stream,
                              void* const* d_in, const int* in_sizes, int n_in,
                              void* d_out, int out_size,
                              const int* in_dtypes, int out_dtype,
                              void* const* d_out_all,
                              void* const* d_ws_all, size_t ws_bytes) {
  (void)comm; (void)in_sizes; (void)in_dtypes; (void)out_dtype; (void)d_out_all;
  if (nranks != NR || (unsigned)rank >= (unsigned)NR || n_in < 4 || ws_bytes < WS_NEED ||
      out_size != NR * M_SH * N_SH) {
    fprintf(stderr, "unexpected problem configuration\n");
    return;
  }
  Params p;
  for (int j = 0; j < 2 * (NR - 1); ++j) p.peer[j] = (char*)d_ws_all[(rank + 1 + j % (NR - 1)) % NR];
  p.ws = (char*)d_ws_all[rank];
  p.x = (const signed char*)d_in[0];
  p.w = (const signed char*)d_in[1];
  p.sx = (const float*)d_in[2];
  p.sw = (const float*)d_in[3];
  p.out = (float*)d_out;
  p.rank = rank;
  hipLaunchKernelGGL(ag_gemm_kernel, dim3(NWG), dim3(512), 0, stream, p);
  HIP_CHECK(hipGetLastError());
}
```
